```python
import numpy as np

kernel_name = "gpu_gemm_rs_m4096_k4096_n4096_f32_silu"


ABSMAX_RTOL = 0.001

COMPUTE_DTYPE = "float32"
EPILOGUE = "silu"
IN_QUANT = None
OUT_QUANT = None
IN_DTYPES = {"x": "float32", "w_mat": "float32"}
WORLD_SIZE = 8

def setup_inputs():

    x = np.random.randn(4096, 4096).astype(np.float32)
    w_mat = (np.random.randn(4096, 4096) / np.sqrt(4096)).astype(np.float32)
    return {"x": x, "w_mat": w_mat}

def _epilogue(y):
    return (y / (1.0 + np.exp(-np.clip(y, -60.0, 60.0)))).astype(np.float32)

def reference(x, w_mat):
    y = (x @ w_mat).astype(np.float32)
    return _epilogue(y)
```

```cpp
#include <hip/hip_runtime.h>
#include <rccl/rccl.h>
#include <cstdio>
#include <cstdint>

#define HIP_CHECK(expr) do { hipError_t _e = (expr); if (_e != hipSuccess) { \
    fprintf(stderr, "HIP error %s at %s:%d\n", hipGetErrorString(_e), __FILE__, __LINE__); \
    return; } } while (0)

typedef unsigned u32x4 __attribute__((ext_vector_type(4)));
typedef unsigned u32x2 __attribute__((ext_vector_type(2)));
typedef float f32x4 __attribute__((ext_vector_type(4)));
typedef float f32x2 __attribute__((ext_vector_type(2)));
typedef _Float16 f16x8 __attribute__((ext_vector_type(8)));
typedef _Float16 f16x2 __attribute__((ext_vector_type(2)));
typedef unsigned long long u64;

namespace {

constexpr int NR = 8;
constexpr int MM = 4096, NN = 4096;
constexpr int KS = 512;
constexpr int NSTEP = 64;

constexpr size_t X_STEP = 16384, X_RB = X_STEP * NSTEP, X_BYTES = 8 * X_RB;
constexpr size_t W_STEP = 8192, W_CG = W_STEP * NSTEP, W_BYTES = 32 * W_CG;
constexpr size_t PAR_BYTES = X_BYTES + W_BYTES;
constexpr size_t OFF_FX = 2 * PAR_BYTES;
constexpr size_t FX_PAR = 8 * 64 * 16;
constexpr size_t OFF_FW = OFF_FX + 2 * FX_PAR;
constexpr size_t FW_PAR = 32 * 64 * 8;
constexpr size_t OFF_FD = OFF_FW + 2 * FW_PAR;
constexpr size_t OFF_TMO = OFF_FD + 1024;
constexpr size_t OFF_CT = OFF_TMO + 1024;
constexpr size_t OFF_DBG = OFF_CT + 256 * 4 * 128;
constexpr size_t WS_NEED = OFF_DBG + 256 * 4 * 64 * 8;

constexpr int GRID = 256, BLOCK = 256;
#define NPUSH_PER_XCD 8
#define XOWN_EARLY 4
constexpr int NPW = NPUSH_PER_XCD * 8 * 4;
constexpr unsigned long long DEADLINE_TICKS = 50000000ull;

constexpr int LDS_RING = 6 * 24576;
constexpr int LDS_BYTES = LDS_RING + 64;

struct Params {
    const float* x;
    const float* w;
    char* ws[NR];
    float* out[NR];
    int rank;
    int pad;
};

__device__ __forceinline__ void store16_sys(void* p, u32x4 v) {
    asm volatile("global_store_dwordx4 %0, %1, off sc0 sc1\n\ts_nop 1" :: "v"(p), "v"(v) : "memory");
}
__device__ __forceinline__ void wait_vm0() { asm volatile("s_waitcnt vmcnt(0)" ::: "memory"); }

__device__ __forceinline__ unsigned pack2(float a, float b) {
    f32x2 v = {a, b};
    return __builtin_bit_cast(unsigned, __builtin_convertvector(v, f16x2));
}

#define STAMP(idx) do {} while (0)

constexpr int U_EARLY = 384, U_LATE = 256;
constexpr int U_A = XOWN_EARLY * U_EARLY;
constexpr int U_B = U_A + (8 - XOWN_EARLY) * U_LATE;
constexpr int U_TOTAL = U_B + (8 - XOWN_EARLY) * 128;

__device__ __forceinline__ void push_role(const Params& p, int pw, unsigned call) {
    const int lane = threadIdx.x & 63;
    const int rank = p.rank;
    const unsigned par = call & 1;
    const int c = lane & 15, kq = lane >> 4;
    for (int u = pw; u < U_TOTAL; u += NPW) {
        int s, kind, j;
        if (u < U_A) { s = u / U_EARLY; int r = u % U_EARLY; kind = r % 3; j = r / 3; }
        else if (u < U_B) { int v = u - U_A; s = XOWN_EARLY + v / U_LATE; int r = v % U_LATE; kind = r & 1; j = r >> 1; }
        else { int v = u - U_B; s = XOWN_EARLY + v / 128; kind = 2; j = v % 128; }
        const int t = rank * 8 + s;
        if (kind == 0) {
            const int g = j & 1, kh = (j >> 1) & 1, cg = j >> 2;
            const float* src = p.w + (size_t)(s * 64 + kh * 32 + 4 * kq) * NN + g * 2048 + cg * 64 + 4 * c;
            f32x4 v[8];
#pragma unroll
            for (int jj = 0; jj < 8; ++jj)
                v[jj] = __builtin_nontemporal_load((const f32x4*)(src + (size_t)((jj & 3) + 16 * (jj >> 2)) * NN));
            u32x4 f[4];
#pragma unroll
            for (int i = 0; i < 4; ++i)
                f[i] = u32x4{pack2(v[0][i], v[1][i]), pack2(v[2][i], v[3][i]), pack2(v[4][i], v[5][i]), pack2(v[6][i], v[7][i])};
            const size_t off = par * PAR_BYTES + X_BYTES + cg * W_CG + t * W_STEP + kh * 4096 + lane * 16;
#pragma unroll
            for (int d = 0; d < 4; ++d) {
                const int q = g * 4 + ((d + rank + cg) & 3);
                char* dst = p.ws[q] + off;
#pragma unroll
                for (int i = 0; i < 4; ++i) store16_sys(dst + i * 1024, f[i]);
            }
            wait_vm0();
            if (lane < 4) {
                unsigned* fl = (unsigned*)(p.ws[g * 4 + lane] + OFF_FW + par * FW_PAR + (cg * 64 + t) * 8 + kh * 4);
                __hip_atomic_store(fl, call, __ATOMIC_RELAXED, __HIP_MEMORY_SCOPE_SYSTEM);
            }
        } else {
            const int q = (j + rank) & 7, ru = j >> 3;
            const int rbl = ru >> 2, sub = ru & 3;
            int row0, rb;
            if (kind == 2) { row0 = q * 512 + rbl * 128 + sub * 32; rb = rbl; }
            else { row0 = ((q & 4) ^ 4) * 512 + rbl * 512 + (q & 3) * 128 + sub * 32; rb = 4 + rbl; }
            const float* src = p.x + (size_t)(row0 + c) * KS + s * 64 + 4 * kq;
            f32x4 v[8];
#pragma unroll
            for (int jj = 0; jj < 8; ++jj)
                v[jj] = *(const f32x4*)(src + (size_t)(jj >> 2) * 16 * KS + (jj & 3) * 16);
            char* dst = p.ws[q] + par * PAR_BYTES + rb * X_RB + t * X_STEP + sub * 4096 + lane * 16;
#pragma unroll
            for (int i = 0; i < 4; ++i) {
                const f32x4 a = v[2 * i], b = v[2 * i + 1];
                store16_sys(dst + i * 1024, u32x4{pack2(a[0], a[1]), pack2(a[2], a[3]), pack2(b[0], b[1]), pack2(b[2], b[3])});
            }
            wait_vm0();
            if (lane == 0) {
                unsigned* fl = (unsigned*)(p.ws[q] + OFF_FX + par * FX_PAR + (rb * 64 + t) * 16 + sub * 4);
                __hip_atomic_store(fl, call, __ATOMIC_RELAXED, __HIP_MEMORY_SCOPE_SYSTEM);
            }
        }
    }
}

__device__ __forceinline__ float silu(float y) {
    const float z = fminf(fmaxf(y, -60.f), 60.f);
    return y * __builtin_amdgcn_rcpf(1.0f + __expf(-z));
}

template <int N> __device__ __forceinline__ void wait_vm() {
    asm volatile("s_waitcnt vmcnt(%0)" :: "i"(N) : "memory");
}

template <int NB>
__device__ __forceinline__ void tile_role(const Params& p, char* lds, int rb, int cg, unsigned call) {
    constexpr int SLOT = 16384 + NB * 8192;
    constexpr int NS = LDS_RING / SLOT;
    constexpr int G = SLOT / 4096;
    const int lane = threadIdx.x & 63;
    const int wave = __builtin_amdgcn_readfirstlane(threadIdx.x >> 6);
    const int rank = p.rank;
    const unsigned par = call & 1;
    char* ws = p.ws[rank];
    const char* xs = ws + par * PAR_BYTES + rb * X_RB + wave * 4096 + lane * 16;
    const char* wsrc = ws + par * PAR_BYTES + X_BYTES + cg * W_CG + (NB == 2 ? (wave >> 1) * W_CG + (wave & 1) * 4096 : wave * 2048) + lane * 16;
    const unsigned* fx = (const unsigned*)(ws + OFF_FX + par * FX_PAR + (rb * 64 + lane) * 16);
    const unsigned* fw = (const unsigned*)(ws + OFF_FW + par * FW_PAR + (cg * 64 + lane) * 8);
    unsigned* tmo = (unsigned*)(ws + OFF_TMO);
    volatile __attribute__((address_space(3))) u64* lmask = (volatile __attribute__((address_space(3))) u64*)(lds + LDS_RING);

    f32x4 acc[2][4 * NB];
#pragma unroll
    for (int m = 0; m < 2; ++m)
#pragma unroll
        for (int n = 0; n < 4 * NB; ++n) acc[m][n] = f32x4{0.f, 0.f, 0.f, 0.f};

    auto issue = [&](int t, int slot) {
        char* l = lds + slot * SLOT;
#pragma unroll
        for (int i = 0; i < 4; ++i)
            __builtin_amdgcn_global_load_lds((const __attribute__((address_space(1))) void*)(xs + (size_t)t * X_STEP + i * 1024),
                (__attribute__((address_space(3))) void*)(l + wave * 4096 + i * 1024), 16, 0, 17);
#pragma unroll
        for (int i = 0; i < 2 * NB; ++i)
            __builtin_amdgcn_global_load_lds((const __attribute__((address_space(1))) void*)(wsrc + (size_t)t * W_STEP + i * 1024),
                (__attribute__((address_space(3))) void*)(l + 16384 + wave * (2048 * NB) + i * 1024), 16, 0, 17);
    };

    u64 done = 0;
    int nb_dbg = 0;
    while (done != ~0ull) {
        if (wave == 0) {
            const u64 t0 = __builtin_amdgcn_s_memrealtime();
            u64 ready;
            for (;;) {
                u32x4 a; u32x2 b, b2;
                if constexpr (NB == 2) {
                    asm volatile("global_load_dwordx4 %0, %3, off sc0 sc1\n\t"
                                 "global_load_dwordx2 %1, %4, off sc0 sc1\n\t"
                                 "global_load_dwordx2 %2, %4, off offset:512 sc0 sc1\n\ts_waitcnt vmcnt(0)"
                                 : "=&v"(a), "=&v"(b), "=&v"(b2) : "v"(fx), "v"(fw) : "memory");
                } else {
                    asm volatile("global_load_dwordx4 %0, %2, off sc0 sc1\n\t"
                                 "global_load_dwordx2 %1, %3, off sc0 sc1\n\ts_waitcnt vmcnt(0)"
                                 : "=&v"(a), "=&v"(b) : "v"(fx), "v"(fw) : "memory");
                    b2 = b;
                }
                const unsigned mn = min(min(min(a[0], a[1]), min(a[2], a[3])), min(min(b[0], b[1]), min(b2[0], b2[1])));
                ready = __builtin_amdgcn_ballot_w64(mn >= call) & ~done;
                if (ready) break;
                if (__hip_atomic_load(tmo, __ATOMIC_RELAXED, __HIP_MEMORY_SCOPE_AGENT) == call ||
                    __builtin_amdgcn_s_memrealtime() - t0 > DEADLINE_TICKS) {
                    __hip_atomic_store(tmo, call, __ATOMIC_RELAXED, __HIP_MEMORY_SCOPE_AGENT);
                    break;
                }
                __builtin_amdgcn_s_sleep(4);
            }
            if (lane == 0) *lmask = ready;
        }
        __syncthreads();
        u64 rd = *lmask;
        rd = ((u64)(unsigned)__builtin_amdgcn_readfirstlane((unsigned)(rd >> 32)) << 32) | (u64)(unsigned)__builtin_amdgcn_readfirstlane((unsigned)rd);
        if (rd == 0) break;
        done |= rd;
        const int n = __builtin_popcountll(rd);
        STAMP(8 + (nb_dbg < 27 ? nb_dbg : 27) * 2);
        int islot = 0;
        for (int j = 0; j < NS - 1 && rd; ++j) {
            const int t = __builtin_ctzll(rd); rd &= rd - 1;
            issue(t, islot); ++islot;
        }
        int cslot = 0;
        for (int j = 0; j < n; ++j) {
            const int rem = n - 1 - j;
            if (rem >= NS - 2) wait_vm<G * (NS - 2)>();
            else if (NS > 5 && rem == 3) wait_vm<G * 3>();
            else if (NS > 4 && rem == 2) wait_vm<G * 2>();
            else if (rem == 1) wait_vm<G>();
            else wait_vm<0>();
            asm volatile("s_waitcnt lgkmcnt(0)" ::: "memory");
            __builtin_amdgcn_s_barrier();
            asm volatile("" ::: "memory");
            if (rd) {
                const int t = __builtin_ctzll(rd); rd &= rd - 1;
                issue(t, islot);
                islot = (islot == NS - 1) ? 0 : islot + 1;
            }
            const char* l = lds + cslot * SLOT + lane * 16;
#pragma unroll
            for (int kh = 0; kh < 2; ++kh) {
                f16x8 a[2], b[4 * NB];
#pragma unroll
                for (int m = 0; m < 2; ++m) a[m] = *(const f16x8*)(l + wave * 4096 + m * 2048 + kh * 1024);
#pragma unroll
                for (int n2 = 0; n2 < 4 * NB; ++n2) b[n2] = *(const f16x8*)(l + 16384 + (n2 >> 2) * 8192 + kh * 4096 + (n2 & 3) * 1024);
#pragma unroll
                for (int m = 0; m < 2; ++m)
#pragma unroll
                    for (int n2 = 0; n2 < 4 * NB; ++n2)
                        acc[m][n2] = __builtin_amdgcn_mfma_f32_16x16x32_f16(a[m], b[n2], acc[m][n2], 0, 0, 0);
            }
            cslot = (cslot == NS - 1) ? 0 : cslot + 1;
        }
        STAMP(9 + (nb_dbg < 27 ? nb_dbg : 27) * 2);
        ++nb_dbg;
        __syncthreads();
    }
    STAMP(1);

    const float bad = __hip_atomic_load(tmo, __ATOMIC_RELAXED, __HIP_MEMORY_SCOPE_AGENT) == call ? __builtin_nanf("") : 0.f;
    const int grp = rank >> 2;
    const int q = rb < 4 ? rank : ((grp ^ 1) * 4 + (rb - 4));
    const int row0 = (rb < 4 ? rb : (rank & 3)) * 128 + wave * 32 + (lane >> 4) * 4;
    float* o = p.out[q] + (size_t)row0 * NN + grp * 2048 + cg * 64 + (lane & 15) * 4;
#pragma unroll
    for (int nbi = 0; nbi < NB; ++nbi)
#pragma unroll
        for (int m = 0; m < 2; ++m)
#pragma unroll
            for (int r = 0; r < 4; ++r) {
                f32x4 v;
#pragma unroll
                for (int i = 0; i < 4; ++i) v[i] = silu(acc[m][nbi * 4 + i][r]) + bad;
                float* dst = o + (size_t)(m * 16 + r) * NN + nbi * 64;
                if (NB == 2) *(f32x4*)dst = v;
                else store16_sys(dst, __builtin_bit_cast(u32x4, v));
            }
    if (NB == 1) {
        wait_vm0();
        __syncthreads();
        if (threadIdx.x == 0) {
            unsigned* fl = (unsigned*)(p.ws[q] + OFF_FD + ((rank & 3) * 32 + cg) * 4);
            __hip_atomic_store(fl, call, __ATOMIC_RELAXED, __HIP_MEMORY_SCOPE_SYSTEM);
        }
    }
    STAMP(2);
}

__global__ void __launch_bounds__(BLOCK, 1) fused_kernel(Params p) {
    extern __shared__ __attribute__((aligned(16))) char lds[];
    const int lane = threadIdx.x & 63;
    const int wave = __builtin_amdgcn_readfirstlane(threadIdx.x >> 6);
    STAMP(0);
    unsigned call = 0;
    if (lane == 0)
        call = __hip_atomic_fetch_add((unsigned*)(p.ws[p.rank] + OFF_CT + (blockIdx.x * 4 + wave) * 128), 1u,
                                      __ATOMIC_RELAXED, __HIP_MEMORY_SCOPE_AGENT) + 1;
    call = __builtin_amdgcn_readfirstlane(call);
    const int xcd = blockIdx.x & 7, lb = blockIdx.x >> 3;
    if (lb < 8) {
        if (lb < NPUSH_PER_XCD) push_role(p, (wave * NPUSH_PER_XCD + lb) * 8 + xcd, call);
        STAMP(1);
        if (blockIdx.x == 0 && wave == 0) {
            const unsigned* fd = (const unsigned*)(p.ws[p.rank] + OFF_FD) + lane * 2;
            unsigned* tmo = (unsigned*)(p.ws[p.rank] + OFF_TMO);
            const u64 t0 = __builtin_amdgcn_s_memrealtime();
            for (;;) {
                u32x2 a;
                asm volatile("global_load_dwordx2 %0, %1, off sc0 sc1\n\ts_waitcnt vmcnt(0)" : "=&v"(a) : "v"(fd) : "memory");
                if (__builtin_amdgcn_ballot_w64(min(a[0], a[1]) >= call) == ~0ull) break;
                if (__hip_atomic_load(tmo, __ATOMIC_RELAXED, __HIP_MEMORY_SCOPE_AGENT) == call ||
                    __builtin_amdgcn_s_memrealtime() - t0 > DEADLINE_TICKS) {
                    __hip_atomic_store(tmo, call, __ATOMIC_RELAXED, __HIP_MEMORY_SCOPE_AGENT);
                    if (lane == 0) p.out[p.rank][((p.rank >> 2) ^ 1) * 2048] = __builtin_nanf("");
                    break;
                }
                __builtin_amdgcn_s_sleep(2);
            }
            STAMP(2);
        }
    } else if (lb < 24) {
        const int k = lb - 8;
        tile_role<1>(p, lds, 4 + (k & 3), xcd * 4 + (k >> 2), call);
    } else {
        const int k = lb - 24;
        tile_role<2>(p, lds, k & 3, xcd * 4 + (k >> 2) * 2, call);
    }
}

}

extern "C" void kernel_launch(int rank, int nranks, ncclComm_t comm,
                              hipStream_t stream,
                              void* const* d_in, const int* in_sizes, int n_in,
                              void* d_out, int out_size,
                              const int* in_dtypes, int out_dtype,
                              void* const* d_out_all,
                              void* const* d_ws_all, size_t ws_bytes) {
    if (nranks != NR || (unsigned)rank >= (unsigned)NR || ws_bytes < WS_NEED || n_in < 2 ||
        in_sizes[0] != MM * KS || in_sizes[1] != KS * NN || out_size != (MM / NR) * NN) {
        fprintf(stderr, "kernel_launch: unexpected shapes\n");
        return;
    }
    HIP_CHECK(hipFuncSetAttribute((const void*)fused_kernel, hipFuncAttributeMaxDynamicSharedMemorySize, LDS_BYTES));
    Params p;
    p.x = (const float*)d_in[0];
    p.w = (const float*)d_in[1];
    for (int i = 0; i < NR; ++i) { p.ws[i] = (char*)d_ws_all[i]; p.out[i] = (float*)d_out_all[i]; }
    p.rank = rank;
    p.pad = 0;
    fused_kernel<<<GRID, BLOCK, LDS_BYTES, stream>>>(p);
    HIP_CHECK(hipGetLastError());
}
```

```cpp
#include <hip/hip_runtime.h>
#include <rccl/rccl.h>
#include <cstdio>
#include <cstdint>

#define HIP_CHECK(expr) do { hipError_t _e = (expr); if (_e != hipSuccess) { \
    fprintf(stderr, "HIP error %s at %s:%d\n", hipGetErrorString(_e), __FILE__, __LINE__); \
    return; } } while (0)

typedef unsigned u32x4 __attribute__((ext_vector_type(4)));
typedef unsigned u32x2 __attribute__((ext_vector_type(2)));
typedef float f32x4 __attribute__((ext_vector_type(4)));
typedef float f32x2 __attribute__((ext_vector_type(2)));
typedef _Float16 f16x8 __attribute__((ext_vector_type(8)));
typedef _Float16 f16x2 __attribute__((ext_vector_type(2)));
typedef unsigned long long u64;

namespace {

constexpr int NR = 8;
constexpr int MM = 4096, NN = 4096;
constexpr int KS = 512;
constexpr int NSTEP = 64;

constexpr size_t X_STEP = 16384, X_RB = X_STEP * NSTEP, X_BYTES = 8 * X_RB;
constexpr size_t W_STEP = 8192, W_CG = W_STEP * NSTEP, W_BYTES = 32 * W_CG;
constexpr size_t PAR_BYTES = X_BYTES + W_BYTES;
constexpr size_t OFF_FX = 2 * PAR_BYTES;
constexpr size_t OFF_FW = OFF_FX + 8 * 64 * 16;
constexpr size_t FL_PAR = 8 * 64 * 16 + 32 * 64 * 8;
constexpr size_t OFF_FD = OFF_FX + 2 * FL_PAR;
constexpr size_t OFF_TMO = OFF_FD + 1024;
constexpr size_t OFF_TK = OFF_TMO + 1024;
constexpr size_t OFF_CT = OFF_TK + 1024;
constexpr size_t OFF_DBG = OFF_CT + 256 * 4 * 128;
constexpr size_t WS_NEED = OFF_DBG + 256 * 4 * 64 * 8;

constexpr int GRID = 256, BLOCK = 256;
#define NPUSH_PER_XCD 8
#define PD 1
#define ROUT_INFLIGHT 6
#define TAU 1942
#define TSTART 150
constexpr int NWP = NPUSH_PER_XCD * 4;
constexpr unsigned long long DEADLINE_TICKS = 50000000ull;

constexpr int LDS_RING = 6 * 24576;
constexpr int LDS_BYTES = LDS_RING + 64;

struct Params {
    const float* x;
    const float* w;
    char* ws[NR];
    float* out[NR];
    int rank;
    int ntab[3];
    unsigned char tab[3 * 64];
};

__device__ __forceinline__ void store16_sys(void* p, u32x4 v) {
    asm volatile("global_store_dwordx4 %0, %1, off sc0 sc1\n\ts_nop 1" :: "v"(p), "v"(v) : "memory");
}
template <int N> __device__ __forceinline__ void wait_vm() {
    asm volatile("s_waitcnt vmcnt(%0)" :: "i"(N) : "memory");
}
__device__ __forceinline__ void wait_vm0() { asm volatile("s_waitcnt vmcnt(0)" ::: "memory"); }

template <class T> __device__ __forceinline__ T* uniform_ptr(T* p) {
    const u64 v = (u64)p;
    return (T*)(((u64)(unsigned)__builtin_amdgcn_readfirstlane((unsigned)(v >> 32)) << 32) | (u64)(unsigned)__builtin_amdgcn_readfirstlane((unsigned)v));
}

__device__ __forceinline__ unsigned now32() { return (unsigned)__builtin_amdgcn_s_memrealtime(); }
__device__ __forceinline__ void wait_until(unsigned t) {
    while ((int)(now32() - t) < 0) __builtin_amdgcn_s_sleep(1);
}

__device__ __forceinline__ unsigned pack2(float a, float b) {
    f32x2 v = {a, b};
    return __builtin_bit_cast(unsigned, __builtin_convertvector(v, f16x2));
}

#define STAMPN(idx, n) do {} while (0)
#define STAMP(idx) do {} while (0)

enum { T_W = 0, T_XR = 1, T_XO = 2, T_RS = 3, T_RF = 4 };

struct Task { int kind; unsigned dst, fl, when; };

__device__ __forceinline__ void push_role(const Params& p, char* lbuf, int e, int jw, unsigned* ctr, unsigned tzero) {
    const int lane = threadIdx.x & 63;
    const int rank = p.rank, q = rank ^ e;
    const int c = lane & 15, kq = lane >> 4;
    const int cls = e == 0 ? 0 : (e < 4 ? 1 : 2);
    int desc = p.tab[cls * 64 + lane];
    const int ntask = p.ntab[cls] * 16;
    char* const dws = uniform_ptr(p.ws[q]);
    char* const lws = p.ws[rank];
    const float* const px = p.x;
    const float* const pw = p.w;
    unsigned call = 0, par = 0;

    auto load = [&](int i, int u, Task& tk) {
        const int d = __builtin_amdgcn_readlane(desc, u >> 4), j = u & 15;
        const int type = d >> 5, s = (d >> 2) & 7, lo2 = d & 3;
        const char* base; unsigned voff, sa, sb;
        tk.when = tzero + (e == 0 ? 0 : (u * TAU) >> 8);
        if (type == T_XR || type == T_XO) {
            const int rbl = j >> 2, sub = j & 3, t = rank * 8 + s;
            const int row0 = type == T_XO ? q * 512 + rbl * 128 + sub * 32
                                          : (((q & 4) ^ 4) + rbl) * 512 + (q & 3) * 128 + sub * 32;
            const int rb = type == T_XO ? rbl : 4 + rbl;
            base = (const char*)(px + (size_t)row0 * KS + s * 64);
            voff = (c * KS + 4 * kq) * 4; sa = 64; sb = 16 * KS * 4;
            tk.kind = 1;
            tk.dst = rb * X_RB + t * X_STEP + sub * 4096;
            tk.fl = OFF_FX + (rb * 64 + t) * 16 + sub * 4;
        } else {
            const int kh = j & 1;
            int cg, t = rank * 8 + s;
            if (type == T_W) cg = (j >> 1) * 4 + lo2;
            else if (type == T_RS) cg = (j >> 3) * 16 + (q & 3) * 4 + ((j >> 1) & 3);
            else { cg = ((j >> 1) & 1) * 16 + (rank & 3) * 4 + lo2; t = (((rank & 4) ^ 4) + (j >> 2)) * 8 + s; }
            tk.dst = X_BYTES + cg * W_CG + t * W_STEP + kh * 4096;
            tk.fl = OFF_FW + (cg * 64 + t) * 8 + kh * 4;
            if (type == T_RF) {
                const unsigned* fl = (const unsigned*)(lws + tk.fl + par * FL_PAR);
                unsigned* tmo = (unsigned*)(lws + OFF_TMO);
                const u64 t0 = __builtin_amdgcn_s_memrealtime();
                while ((unsigned)__builtin_amdgcn_readfirstlane(__hip_atomic_load(fl, __ATOMIC_RELAXED, __HIP_MEMORY_SCOPE_SYSTEM)) < call) {
                    if ((unsigned)__builtin_amdgcn_readfirstlane(__hip_atomic_load(tmo, __ATOMIC_RELAXED, __HIP_MEMORY_SCOPE_AGENT)) == call ||
                        __builtin_amdgcn_s_memrealtime() - t0 > DEADLINE_TICKS) {
                        __hip_atomic_store(tmo, call, __ATOMIC_RELAXED, __HIP_MEMORY_SCOPE_AGENT);
                        tk.fl = OFF_TMO + 128 - par * FL_PAR;
                        break;
                    }
                    __builtin_amdgcn_s_sleep(2);
                }
                base = lws + par * PAR_BYTES + tk.dst;
                voff = lane * 16; sa = 1024; sb = 0;
                tk.kind = 2;
            } else {
                base = (const char*)(pw + (size_t)(s * 64 + kh * 32) * NN + (q >> 2) * 2048 + cg * 64);
                voff = (4 * kq * NN + 4 * c) * 4; sa = NN * 4; sb = 16 * NN * 4;
                tk.kind = 0;
            }
        }
        char* l = lbuf + (i & 1) * 8192;
#pragma unroll
        for (int jj = 0; jj < 8; ++jj)
            if (jj < 4 || tk.kind != 2)
                __builtin_amdgcn_global_load_lds((const __attribute__((address_space(1))) void*)(base + (voff + (jj & 3) * sa + (jj >> 2) * sb)),
                    (__attribute__((address_space(3))) void*)(l + jj * 1024), 16, 0, 17);
        asm volatile("" ::: "memory");
    };
    auto store = [&](int i, const Task& tk) {
        u32x4 v[8];
#pragma unroll
        for (int jj = 0; jj < 8; ++jj) v[jj] = *(const u32x4*)(lbuf + (i & 1) * 8192 + jj * 1024 + lane * 16);
        u32x4 f[4];
        auto fl = [](unsigned x) { return __builtin_bit_cast(float, x); };
        if (tk.kind == 0) {
#pragma unroll
            for (int i = 0; i < 4; ++i)
                f[i] = u32x4{pack2(fl(v[0][i]), fl(v[1][i])), pack2(fl(v[2][i]), fl(v[3][i])),
                             pack2(fl(v[4][i]), fl(v[5][i])), pack2(fl(v[6][i]), fl(v[7][i]))};
        } else if (tk.kind == 1) {
#pragma unroll
            for (int i = 0; i < 4; ++i)
                f[i] = u32x4{pack2(fl(v[2 * i][0]), fl(v[2 * i][1])), pack2(fl(v[2 * i][2]), fl(v[2 * i][3])),
                             pack2(fl(v[2 * i + 1][0]), fl(v[2 * i + 1][1])), pack2(fl(v[2 * i + 1][2]), fl(v[2 * i + 1][3]))};
        } else {
#pragma unroll
            for (int i = 0; i < 4; ++i) f[i] = v[i];
        }
        wait_until(tk.when);
        asm volatile("global_store_dwordx4 %0, %1, %5 sc0 sc1\n\t"
                     "global_store_dwordx4 %0, %2, %5 offset:1024 sc0 sc1\n\t"
                     "global_store_dwordx4 %0, %3, %5 offset:2048 sc0 sc1\n\t"
                     "global_store_dwordx4 %0, %4, %5 offset:3072 sc0 sc1\n\ts_nop 1"
                     :: "v"(lane * 16 + par * (unsigned)PAR_BYTES + tk.dst), "v"(f[0]), "v"(f[1]), "v"(f[2]), "v"(f[3]), "s"(dws) : "memory");
    };
    auto flag = [&](unsigned off) {
        if (lane == 0)
            asm volatile("global_store_dword %0, %1, %2 sc0 sc1" :: "v"(off + par * (unsigned)FL_PAR), "v"(call), "s"(dws) : "memory");
    };

    unsigned* const tick = (unsigned*)(lws + OFF_TK + e * 128);
    asm volatile("" : "+v"(desc) :: "memory");
    Task t0 = {0, 0, 0, 0};
    int un = jw;
    unsigned pfl = 0;
    bool have = false;
#pragma unroll 1
    for (int i = -1;; ++i) {
        if (i >= 1) flag(pfl);
        if (i >= 0 && !have) break;
        if (i >= 0) { store(i, t0); pfl = t0.fl; }
        STAMPN(8 + min(i + 1, 55), un);
        have = un < ntask;
        unsigned cv = 0;
        if (have) {
            load(i + 1, un, t0);
            if (lane == 0) asm volatile("global_atomic_add %0, %1, %2, off sc0" : "=v"(cv) : "v"(i < 0 ? ctr : tick), "v"(1u) : "memory");
        }
        asm volatile("s_waitcnt vmcnt(0)" : "+v"(cv) :: "memory");
        const unsigned tv = __builtin_amdgcn_readfirstlane(cv);
        if (i < 0) { call = tv + 1; par = call & 1; un = NWP + jw; }
        else un = (int)min(tv - (call - 1) * (unsigned)(ntask - NWP) + 2 * NWP, (unsigned)ntask);
    }
    STAMP(1);
    if (blockIdx.x == 0 && e == 0) {
        const unsigned* fd = (const unsigned*)(lws + OFF_FD) + lane;
        unsigned* tmo = (unsigned*)(lws + OFF_TMO);
        const u64 t0 = __builtin_amdgcn_s_memrealtime();
        for (;;) {
            const unsigned a = __hip_atomic_load(fd, __ATOMIC_RELAXED, __HIP_MEMORY_SCOPE_SYSTEM);
            if (__builtin_amdgcn_ballot_w64(a >= call) == ~0ull) break;
            if (__hip_atomic_load(tmo, __ATOMIC_RELAXED, __HIP_MEMORY_SCOPE_AGENT) == call ||
                __builtin_amdgcn_s_memrealtime() - t0 > DEADLINE_TICKS) {
                __hip_atomic_store(tmo, call, __ATOMIC_RELAXED, __HIP_MEMORY_SCOPE_AGENT);
                if (lane == 0) p.out[rank][((rank >> 2) ^ 1) * 2048] = __builtin_nanf("");
                break;
            }
            __builtin_amdgcn_s_sleep(2);
        }
        STAMP(2);
    }
}

__device__ __forceinline__ float silu(float y) {
    const float z = fminf(fmaxf(y, -60.f), 60.f);
    return y * __builtin_amdgcn_rcpf(1.0f + __expf(-z));
}

template <int NB>
__device__ __forceinline__ void tile_role(const Params& p, char* lds, int rb, int cg, unsigned call, unsigned tzero) {
    constexpr int SLOT = 16384 + NB * 8192;
    constexpr int NS = LDS_RING / SLOT;
    constexpr int G = SLOT / 4096;
    const int lane = threadIdx.x & 63;
    const int wave = __builtin_amdgcn_readfirstlane(threadIdx.x >> 6);
    const int rank = p.rank;
    const unsigned par = call & 1;
    char* ws = p.ws[rank];
    const char* xs = ws + par * PAR_BYTES + rb * X_RB + wave * 4096 + lane * 16;
    const char* wsrc = ws + par * PAR_BYTES + X_BYTES + cg * W_CG + (NB == 2 ? (wave >> 1) * W_CG + (wave & 1) * 4096 : wave * 2048) + lane * 16;
    const unsigned* fx = (const unsigned*)(ws + OFF_FX + par * FL_PAR + (rb * 64 + lane) * 16);
    const unsigned* fw = (const unsigned*)(ws + OFF_FW + par * FL_PAR + (cg * 64 + lane) * 8);
    unsigned* tmo = (unsigned*)(ws + OFF_TMO);
    volatile __attribute__((address_space(3))) u64* lmask = (volatile __attribute__((address_space(3))) u64*)(lds + LDS_RING);

    f32x4 acc[2][4 * NB];
#pragma unroll
    for (int m = 0; m < 2; ++m)
#pragma unroll
        for (int n = 0; n < 4 * NB; ++n) acc[m][n] = f32x4{0.f, 0.f, 0.f, 0.f};

    auto issue = [&](int t, int slot) {
        char* l = lds + slot * SLOT;
#pragma unroll
        for (int i = 0; i < 4; ++i)
            __builtin_amdgcn_global_load_lds((const __attribute__((address_space(1))) void*)(xs + (size_t)t * X_STEP + i * 1024),
                (__attribute__((address_space(3))) void*)(l + wave * 4096 + i * 1024), 16, 0, 17);
#pragma unroll
        for (int i = 0; i < 2 * NB; ++i)
            __builtin_amdgcn_global_load_lds((const __attribute__((address_space(1))) void*)(wsrc + (size_t)t * W_STEP + i * 1024),
                (__attribute__((address_space(3))) void*)(l + 16384 + wave * (2048 * NB) + i * 1024), 16, 0, 17);
    };

    u64 done = 0;
    int nb_dbg = 0;
    while (done != ~0ull) {
        if (wave == 0) {
            const u64 t0 = __builtin_amdgcn_s_memrealtime();
            u64 ready;
            for (;;) {
                u32x4 a; u32x2 b, b2;
                if constexpr (NB == 2) {
                    asm volatile("global_load_dwordx4 %0, %3, off sc0 sc1\n\t"
                                 "global_load_dwordx2 %1, %4, off sc0 sc1\n\t"
                                 "global_load_dwordx2 %2, %4, off offset:512 sc0 sc1\n\ts_waitcnt vmcnt(0)"
                                 : "=&v"(a), "=&v"(b), "=&v"(b2) : "v"(fx), "v"(fw) : "memory");
                } else {
                    asm volatile("global_load_dwordx4 %0, %2, off sc0 sc1\n\t"
                                 "global_load_dwordx2 %1, %3, off sc0 sc1\n\ts_waitcnt vmcnt(0)"
                                 : "=&v"(a), "=&v"(b) : "v"(fx), "v"(fw) : "memory");
                    b2 = b;
                }
                const unsigned mn = min(min(min(a[0], a[1]), min(a[2], a[3])), min(min(b[0], b[1]), min(b2[0], b2[1])));
                ready = __builtin_amdgcn_ballot_w64(mn >= call) & ~done;
                if (ready) break;
                if (__hip_atomic_load(tmo, __ATOMIC_RELAXED, __HIP_MEMORY_SCOPE_AGENT) == call ||
                    __builtin_amdgcn_s_memrealtime() - t0 > DEADLINE_TICKS) {
                    __hip_atomic_store(tmo, call, __ATOMIC_RELAXED, __HIP_MEMORY_SCOPE_AGENT);
                    break;
                }
                __builtin_amdgcn_s_sleep(4);
            }
            if (lane == 0) *lmask = ready;
        }
        __syncthreads();
        u64 rd = *lmask;
        rd = ((u64)(unsigned)__builtin_amdgcn_readfirstlane((unsigned)(rd >> 32)) << 32) | (u64)(unsigned)__builtin_amdgcn_readfirstlane((unsigned)rd);
        if (rd == 0) break;
        done |= rd;
        const int n = __builtin_popcountll(rd);
        STAMPN(8 + (nb_dbg < 27 ? nb_dbg : 27) * 2, n);
        int islot = 0;
        for (int j = 0; j < NS - 1 && rd; ++j) {
            const int t = __builtin_ctzll(rd); rd &= rd - 1;
            issue(t, islot); ++islot;
        }
        int cslot = 0;
        for (int j = 0; j < n; ++j) {
            const int rem = n - 1 - j;
            if (rem >= NS - 2) wait_vm<G * (NS - 2)>();
            else if (NS > 5 && rem == 3) wait_vm<G * 3>();
            else if (NS > 4 && rem == 2) wait_vm<G * 2>();
            else if (rem == 1) wait_vm<G>();
            else wait_vm<0>();
            asm volatile("s_waitcnt lgkmcnt(0)" ::: "memory");
            __builtin_amdgcn_s_barrier();
            asm volatile("" ::: "memory");
            if (rd) {
                const int t = __builtin_ctzll(rd); rd &= rd - 1;
                issue(t, islot);
                islot = (islot == NS - 1) ? 0 : islot + 1;
            }
            const char* l = lds + cslot * SLOT + lane * 16;
#pragma unroll
            for (int kh = 0; kh < 2; ++kh) {
                f16x8 a[2], b[4 * NB];
#pragma unroll
                for (int m = 0; m < 2; ++m) a[m] = *(const f16x8*)(l + wave * 4096 + m * 2048 + kh * 1024);
#pragma unroll
                for (int n2 = 0; n2 < 4 * NB; ++n2) b[n2] = *(const f16x8*)(l + 16384 + (n2 >> 2) * 8192 + kh * 4096 + (n2 & 3) * 1024);
#pragma unroll
                for (int m = 0; m < 2; ++m)
#pragma unroll
                    for (int n2 = 0; n2 < 4 * NB; ++n2)
                        acc[m][n2] = __builtin_amdgcn_mfma_f32_16x16x32_f16(a[m], b[n2], acc[m][n2], 0, 0, 0);
            }
            cslot = (cslot == NS - 1) ? 0 : cslot + 1;
        }
        STAMP(9 + (nb_dbg < 27 ? nb_dbg : 27) * 2);
        ++nb_dbg;
        __syncthreads();
    }
    STAMP(1);

    const float bad = __hip_atomic_load(tmo, __ATOMIC_RELAXED, __HIP_MEMORY_SCOPE_AGENT) == call ? __builtin_nanf("") : 0.f;
    const int grp = rank >> 2;
    const int q = rb < 4 ? rank : ((grp ^ 1) * 4 + (rb - 4));
    const int row0 = (rb < 4 ? rb : (rank & 3)) * 128 + wave * 32 + (lane >> 4) * 4;
    float* o = p.out[q] + (size_t)row0 * NN + grp * 2048 + cg * 64 + (lane & 15) * 4;
    unsigned when = tzero + (((p.ntab[2] * 16 + ((cg >> 1) & 1) * 128) * 4 + (cg >> 2) * 4 + wave) * TAU >> 10);
#pragma unroll
    for (int nbi = 0; nbi < NB; ++nbi)
#pragma unroll
        for (int m = 0; m < 2; ++m)
#pragma unroll
            for (int r = 0; r < 4; ++r) {
                f32x4 v;
#pragma unroll
                for (int i = 0; i < 4; ++i) v[i] = silu(acc[m][nbi * 4 + i][r]) + bad;
                float* dst = o + (size_t)(m * 16 + r) * NN + nbi * 64;
                if (NB == 2) {
                    wait_until(when);
                    const unsigned t = now32();
                    when = ((int)(t - when) > (int)(8 * TAU >> 8) ? t : when) + (8 * TAU >> 8);
                }
                store16_sys(dst, __builtin_bit_cast(u32x4, v));
                if (NB == 2) wait_vm<ROUT_INFLIGHT>();
            }
    if (NB == 2) {
        wait_vm0();
        __syncthreads();
        if (threadIdx.x == 0) {
            unsigned* fl = (unsigned*)(p.ws[q] + OFF_FD + ((rank & 3) * 16 + (cg >> 1)) * 4);
            __hip_atomic_store(fl, call, __ATOMIC_RELAXED, __HIP_MEMORY_SCOPE_SYSTEM);
        }
    }
    STAMP(2);
}

__global__ void __launch_bounds__(BLOCK, 1) fused_kernel(Params p) {
    extern __shared__ __attribute__((aligned(16))) char lds[];
    const int lane = threadIdx.x & 63;
    const int wave = __builtin_amdgcn_readfirstlane(threadIdx.x >> 6);
    STAMP(0);
    const unsigned tzero = now32() + TSTART;
    unsigned* ctr = (unsigned*)(p.ws[p.rank] + OFF_CT + (blockIdx.x * 4 + wave) * 128);
    const int xcd = blockIdx.x & 7, lb = blockIdx.x >> 3;
    if (lb < 8) {
        if (lb < NPUSH_PER_XCD) push_role(p, lds + wave * 16384, wave * 2 + (lb & 1), (lb >> 1) * 8 + xcd, ctr, tzero);
        return;
    }
    unsigned call = 0;
    if (lane == 0) call = __hip_atomic_fetch_add(ctr, 1u, __ATOMIC_RELAXED, __HIP_MEMORY_SCOPE_AGENT) + 1;
    call = __builtin_amdgcn_readfirstlane(call);
    if (lb < 24) {
        const int k = lb - 8;
        tile_role<1>(p, lds, k & 3, xcd * 4 + (k >> 2), call, tzero);
    } else {
        const int k = lb - 24;
        tile_role<2>(p, lds, 4 + (k & 3), xcd * 4 + (k >> 2) * 2, call, tzero);
    }
}

struct Tab { int n[3]; unsigned char d[3][64]; };
inline Tab make_tab() {
    Tab t{};
    auto add = [&](int cls, int type, int s, int lo2) { t.d[cls][t.n[cls]++] = (unsigned char)(type << 5 | s << 2 | lo2); };
    for (int s = 0; s < 8; ++s) { for (int l = 0; l < 4; ++l) add(0, T_W, s, l); add(0, T_XR, s, 0); add(0, T_XO, s, 0); }
    for (int s = 0; s < 8; ++s) {
        add(1, T_W, s, 0); add(1, T_W, s, 1); add(1, T_XR, s, 0);
        if (s & 1) { add(1, T_W, s >> 1, 2); add(1, T_W, s >> 1, 3); }
        if (s >= 2 && s < 6) { add(1, T_RF, s & 1, (s - 2) & 2); add(1, T_RF, s & 1, ((s - 2) & 2) + 1); }
    }
    for (int s = 4; s < 8; ++s) { add(1, T_W, s, 2); add(1, T_W, s, 3); }
    for (int s = 0; s < 8; ++s) add(1, T_XO, s, 0);
    add(2, T_RS, 0, 0); add(2, T_RS, 1, 0);
    add(2, T_XR, 0, 0); add(2, T_XR, 1, 0); add(2, T_XO, 0, 0); add(2, T_XO, 1, 0);
    for (int s = 2; s < 8; ++s) {
        for (int l = 0; l < (s < 7 ? 4 : 2); ++l) add(2, T_W, s, l);
        add(2, T_XR, s, 0);
        if (s < 7) add(2, T_XO, s, 0);
    }
    add(2, T_W, 7, 2); add(2, T_W, 7, 3); add(2, T_XO, 7, 0);
    return t;
}

}

extern "C" void kernel_launch(int rank, int nranks, ncclComm_t comm,
                              hipStream_t stream,
                              void* const* d_in, const int* in_sizes, int n_in,
                              void* d_out, int out_size,
                              const int* in_dtypes, int out_dtype,
                              void* const* d_out_all,
                              void* const* d_ws_all, size_t ws_bytes) {
    if (nranks != NR || (unsigned)rank >= (unsigned)NR || ws_bytes < WS_NEED || n_in < 2 ||
        in_sizes[0] != MM * KS || in_sizes[1] != KS * NN || out_size != (MM / NR) * NN) {
        fprintf(stderr, "kernel_launch: unexpected shapes\n");
        return;
    }
    HIP_CHECK(hipFuncSetAttribute((const void*)fused_kernel, hipFuncAttributeMaxDynamicSharedMemorySize, LDS_BYTES));
    Params p;
    p.x = (const float*)d_in[0];
    p.w = (const float*)d_in[1];
    for (int i = 0; i < NR; ++i) { p.ws[i] = (char*)d_ws_all[i]; p.out[i] = (float*)d_out_all[i]; }
    p.rank = rank;
    const Tab tab = make_tab();
    for (int i = 0; i < 3; ++i) { p.ntab[i] = tab.n[i]; for (int j = 0; j < 64; ++j) p.tab[i * 64 + j] = tab.d[i][j]; }
    fused_kernel<<<GRID, BLOCK, LDS_BYTES, stream>>>(p);
    HIP_CHECK(hipGetLastError());
}
```

```cpp
#include <hip/hip_runtime.h>
#include <rccl/rccl.h>
#include <cstdio>
#include <cstdint>

#define HIP_CHECK(expr) do { hipError_t _e = (expr); if (_e != hipSuccess) { \
    fprintf(stderr, "HIP error %s at %s:%d\n", hipGetErrorString(_e), __FILE__, __LINE__); \
    return; } } while (0)

typedef unsigned u32x4 __attribute__((ext_vector_type(4)));
typedef unsigned u32x2 __attribute__((ext_vector_type(2)));
typedef float f32x4 __attribute__((ext_vector_type(4)));
typedef float f32x2 __attribute__((ext_vector_type(2)));
typedef _Float16 f16x8 __attribute__((ext_vector_type(8)));
typedef _Float16 f16x2 __attribute__((ext_vector_type(2)));
typedef unsigned long long u64;

namespace {

constexpr int NR = 8;
constexpr int MM = 4096, NN = 4096;
constexpr int KS = 512;
constexpr int NSTEP = 64;

constexpr size_t X_STEP = 16384, X_RB = X_STEP * NSTEP, X_BYTES = 8 * X_RB;
constexpr size_t W_STEP = 8192, W_CG = W_STEP * NSTEP, W_BYTES = 32 * W_CG;
constexpr size_t PAR_BYTES = X_BYTES + W_BYTES;
constexpr size_t OFF_FX = 2 * PAR_BYTES;
constexpr size_t OFF_FW = OFF_FX + 8 * 64 * 16;
constexpr size_t FL_PAR = 8 * 64 * 16 + 32 * 64 * 8;
constexpr size_t OFF_FD = OFF_FX + 2 * FL_PAR;
constexpr size_t OFF_TMO = OFF_FD + 1024;
constexpr size_t OFF_TK = OFF_TMO + 1024;
constexpr size_t OFF_HO = OFF_TK + 1024;
constexpr size_t OFF_CT = OFF_HO + 4096;
constexpr size_t OFF_DBG = OFF_CT + 256 * 128;
constexpr size_t WS_NEED = OFF_DBG + 256 * 8 * 64 * 8;

constexpr int GRID = 256, BLOCK = 512;
#define ROUT_INFLIGHT 6
#define TAU 1942
#define CREDIT 60
#define IN_AUX 0
constexpr int NWP = 64;
constexpr unsigned long long DEADLINE_TICKS = 50000000ull;

constexpr int LDS_RING = 6 * 24576;
constexpr int LDS_BYTES = LDS_RING + 512;
typedef volatile __attribute__((address_space(3))) unsigned* ldsw_t;
typedef __attribute__((address_space(3))) void* ldsp_t;
typedef const __attribute__((address_space(1))) void* glbp_t;

struct Params {
    const float* x;
    const float* w;
    char* ws[NR];
    float* out[NR];
    int rank;
    int ntab[3];
    unsigned char tab[3 * 64];
};

__device__ __forceinline__ void store16_sys(void* p, u32x4 v) {
    asm volatile("global_store_dwordx4 %0, %1, off sc0 sc1\n\ts_nop 1" :: "v"(p), "v"(v) : "memory");
}
template <int N> __device__ __forceinline__ void wait_vm() {
    asm volatile("s_waitcnt vmcnt(%0)" :: "i"(N) : "memory");
}
__device__ __forceinline__ void wait_vm0() { asm volatile("s_waitcnt vmcnt(0)" ::: "memory"); }

template <class T> __device__ __forceinline__ T* uniform_ptr(T* p) {
    const u64 v = (u64)p;
    return (T*)(((u64)(unsigned)__builtin_amdgcn_readfirstlane((unsigned)(v >> 32)) << 32) | (u64)(unsigned)__builtin_amdgcn_readfirstlane((unsigned)v));
}

__device__ __forceinline__ unsigned now32() { return (unsigned)__builtin_amdgcn_s_memrealtime(); }
__device__ __forceinline__ void wait_until(unsigned t) {
    while ((int)(now32() - t) < 0) __builtin_amdgcn_s_sleep(1);
}

__device__ __forceinline__ unsigned pack2(float a, float b) {
    f32x2 v = {a, b};
    return __builtin_bit_cast(unsigned, __builtin_convertvector(v, f16x2));
}

#define STAMPN(idx, n) do {} while (0)
#define STAMP(idx) do {} while (0)

__device__ __forceinline__ unsigned lds_get(ldsw_t w) { return __builtin_amdgcn_readfirstlane(*w); }
__device__ __forceinline__ void lds_wait(ldsw_t w, unsigned v) {
    while (lds_get(w) < v) __builtin_amdgcn_s_sleep(1);
    asm volatile("" ::: "memory");
}
__device__ __forceinline__ void glds4k(const char* g, char* l) {
    __builtin_amdgcn_global_load_lds((glbp_t)g, (ldsp_t)l, 16, 0, 17);
    __builtin_amdgcn_global_load_lds((glbp_t)g, (ldsp_t)l, 16, 1024, 17);
    __builtin_amdgcn_global_load_lds((glbp_t)g, (ldsp_t)l, 16, 2048, 17);
    __builtin_amdgcn_global_load_lds((glbp_t)g, (ldsp_t)l, 16, 3072, 17);
}
__device__ __forceinline__ unsigned lds_add(ldsw_t w, unsigned v) {
    unsigned r; u64 sv;
    asm volatile("s_mov_b64 %1, exec\n\ts_mov_b64 exec, 1\n\tds_add_rtn_u32 %0, %2, %3\n\ts_mov_b64 exec, %1\n\ts_waitcnt lgkmcnt(0)"
                 : "=&v"(r), "=&s"(sv) : "v"(w), "v"(v) : "memory");
    return __builtin_amdgcn_readfirstlane(r);
}

enum { T_W = 0, T_XR = 1, T_XO = 2, T_RS = 3, T_RF = 4 };

struct Task { int kind; unsigned dst, fl; };

__device__ __forceinline__ void push_role(const Params& p, char* lds, int e, int xcd, unsigned* ctr) {
    const int lane = threadIdx.x & 63;
    const int wave = __builtin_amdgcn_readfirstlane(threadIdx.x >> 6);
    const int jw = wave * 8 + xcd;
    char* const lbuf = lds + wave * 16384;
    const ldsw_t ctl = (ldsw_t)(lds + LDS_RING);
    const int rank = p.rank, q = rank ^ e;
    const int c = lane & 15, kq = lane >> 4;
    const int cls = e == 0 ? 0 : (e < 4 ? 1 : 2);
    int desc = p.tab[cls * 64 + lane];
    const int ntask = p.ntab[cls] * 16;
    char* const dws = uniform_ptr(p.ws[q]);
    char* const lws = p.ws[rank];
    const float* const px = p.x;
    const float* const pw = p.w;
    unsigned call = 0, par = 0, tzero = 0;

    auto load = [&](int i, int u, Task& tk) {
        const int d = __builtin_amdgcn_readlane(desc, u >> 4), j = u & 15;
        const int type = d >> 5, s = (d >> 2) & 7, lo2 = d & 3;
        const char* base; unsigned voff, sa, sb;
        if (type == T_XR || type == T_XO) {
            const int rbl = j >> 2, sub = j & 3, t = rank * 8 + s;
            const int row0 = type == T_XO ? q * 512 + rbl * 128 + sub * 32
                                          : (((q & 4) ^ 4) + rbl) * 512 + (q & 3) * 128 + sub * 32;
            const int rb = type == T_XO ? rbl : 4 + rbl;
            base = (const char*)(px + (size_t)row0 * KS + s * 64);
            voff = (c * KS + 4 * kq) * 4; sa = 64; sb = 16 * KS * 4;
            tk.kind = 1;
            tk.dst = rb * X_RB + t * X_STEP + sub * 4096;
            tk.fl = OFF_FX + (rb * 64 + t) * 16 + sub * 4;
        } else {
            const int kh = j & 1;
            int cg, t = rank * 8 + s;
            if (type == T_W) cg = (j >> 1) * 4 + lo2;
            else if (type == T_RS) cg = (j >> 3) * 16 + (q & 3) * 4 + ((j >> 1) & 3);
            else { cg = ((j >> 1) & 1) * 16 + (rank & 3) * 4 + lo2; t = (((rank & 4) ^ 4) + (j >> 2)) * 8 + s; }
            tk.dst = X_BYTES + cg * W_CG + t * W_STEP + kh * 4096;
            tk.fl = OFF_FW + (cg * 64 + t) * 8 + kh * 4;
            base = (const char*)(pw + (size_t)(s * 64 + kh * 32) * NN + (q >> 2) * 2048 + cg * 64);
            voff = (4 * kq * NN + 4 * c) * 4; sa = NN * 4; sb = 16 * NN * 4;
            tk.kind = type == T_RF ? 2 : 0;
        }
        if (tk.kind != 2) {
            char* l = lbuf + (i & 1) * 8192;
#pragma unroll
            for (int jj = 0; jj < 8; ++jj)
                __builtin_amdgcn_global_load_lds((glbp_t)(base + (voff + (jj & 3) * sa + (jj >> 2) * sb)), (ldsp_t)(l + jj * 1024), 16, 0, IN_AUX);
        }
        asm volatile("" ::: "memory");
    };
    auto load_relay = [&](int i, Task& tk) {
        const unsigned* fl = (const unsigned*)(lws + tk.fl + par * FL_PAR);
        unsigned* tmo = (unsigned*)(lws + OFF_TMO);
        const u64 t0 = __builtin_amdgcn_s_memrealtime();
        while ((unsigned)__builtin_amdgcn_readfirstlane(__hip_atomic_load(fl, __ATOMIC_RELAXED, __HIP_MEMORY_SCOPE_SYSTEM)) < call) {
            if ((unsigned)__builtin_amdgcn_readfirstlane(__hip_atomic_load(tmo, __ATOMIC_RELAXED, __HIP_MEMORY_SCOPE_AGENT)) == call ||
                __builtin_amdgcn_s_memrealtime() - t0 > DEADLINE_TICKS) {
                __hip_atomic_store(tmo, call, __ATOMIC_RELAXED, __HIP_MEMORY_SCOPE_AGENT);
                tk.fl = OFF_TMO + 128 - par * FL_PAR;
                break;
            }
            __builtin_amdgcn_s_sleep(2);
        }
        asm volatile("" ::: "memory");
        glds4k(lws + par * PAR_BYTES + tk.dst + lane * 16, lbuf + (i & 1) * 8192);
        wait_vm0();
    };
    auto store = [&](int i, const Task& tk, bool last) {
        u32x4 v[8];
#pragma unroll
        for (int jj = 0; jj < 8; ++jj) v[jj] = *(const u32x4*)(lbuf + (i & 1) * 8192 + jj * 1024 + lane * 16);
        u32x4 f[4];
        auto fl = [](unsigned x) { return __builtin_bit_cast(float, x); };
        if (tk.kind == 0) {
#pragma unroll
            for (int i = 0; i < 4; ++i)
                f[i] = u32x4{pack2(fl(v[0][i]), fl(v[1][i])), pack2(fl(v[2][i]), fl(v[3][i])),
                             pack2(fl(v[4][i]), fl(v[5][i])), pack2(fl(v[6][i]), fl(v[7][i]))};
        } else if (tk.kind == 1) {
#pragma unroll
            for (int i = 0; i < 4; ++i)
                f[i] = u32x4{pack2(fl(v[2 * i][0]), fl(v[2 * i][1])), pack2(fl(v[2 * i][2]), fl(v[2 * i][3])),
                             pack2(fl(v[2 * i + 1][0]), fl(v[2 * i + 1][1])), pack2(fl(v[2 * i + 1][2]), fl(v[2 * i + 1][3]))};
        } else {
#pragma unroll
            for (int i = 0; i < 4; ++i) f[i] = v[i];
        }
        if (e != 0) {
            unsigned slot; u64 sv;
            asm volatile("s_mov_b64 %1, exec\n\ts_mov_b64 exec, 1\n\tds_max_u32 %2, %3\n\tds_add_rtn_u32 %0, %2, %4\n\t"
                         "s_mov_b64 exec, %1\n\ts_waitcnt lgkmcnt(0)"
                         : "=&v"(slot), "=&s"(sv) : "v"(ctl + 2), "v"((now32() - tzero) << 5), "v"(TAU) : "memory");
            if (last && e >= 4 && lds_add(ctl + 3, 1) == 7) {
                const u32x2 ho = {tzero - CREDIT + (lds_get(ctl + 2) >> 5), call};
                if (lane == 0)
                    asm volatile("global_store_dwordx2 %0, %1, off sc0 sc1\n\ts_nop 1" :: "v"(lws + OFF_HO + ((e - 4) * 8 + xcd) * 64), "v"(ho) : "memory");
            }
            wait_until(tzero - CREDIT + ((unsigned)__builtin_amdgcn_readfirstlane(slot) >> 5));
        }
        asm volatile("global_store_dwordx4 %0, %1, %5 sc0 sc1\n\t"
                     "global_store_dwordx4 %0, %2, %5 offset:1024 sc0 sc1\n\t"
                     "global_store_dwordx4 %0, %3, %5 offset:2048 sc0 sc1\n\t"
                     "global_store_dwordx4 %0, %4, %5 offset:3072 sc0 sc1\n\ts_nop 1"
                     :: "v"(lane * 16 + par * (unsigned)PAR_BYTES + tk.dst), "v"(f[0]), "v"(f[1]), "v"(f[2]), "v"(f[3]), "s"(dws) : "memory");
    };

    unsigned* const tick = (unsigned*)(lws + OFF_TK + e * 128);
    asm volatile("" : "+v"(desc) :: "memory");
    Task cur = {0, 0, 0}, nxt = {0, 0, 0};
    int un = jw;
#pragma unroll 1
    for (int i = -1;; ++i) {
        cur = nxt;
        const bool have = un < ntask;
        if (have) load(i + 1, un, nxt);
        unsigned cv; u64 sv;
        asm volatile("s_mov_b64 %1, exec\n\ts_mov_b64 exec, 1\n\tglobal_atomic_add %0, %2, %3, off sc0\n\ts_mov_b64 exec, %1"
                     : "=&v"(cv), "=&s"(sv) : "v"(i < 0 ? ctr : tick), "v"((i < 0 ? wave == 0 : have) ? 1u : 0u) : "memory");
        if (i >= 0) store(i, cur, !have);
        STAMPN(8 + min(i + 1, 55), un);
        asm volatile("s_waitcnt vmcnt(0)" : "+v"(cv) :: "memory");
        const unsigned tv = __builtin_amdgcn_readfirstlane(cv);
        if (i >= 0) {
            if (lane == 0)
                asm volatile("global_store_dword %0, %1, %2 sc0 sc1" :: "v"(cur.fl + par * (unsigned)FL_PAR), "v"(call), "s"(dws) : "memory");
            if (!have) break;
            un = (int)min(tv - (call - 1) * (unsigned)(ntask - NWP) + 2 * NWP, (unsigned)ntask);
        } else {
            if (threadIdx.x == 0) { ctl[0] = tv + 1; ctl[1] = now32(); ctl[2] = 0; ctl[3] = 0; }
            __syncthreads();
            call = lds_get(ctl); tzero = lds_get(ctl + 1); par = call & 1;
            un = NWP + jw;
        }
        if (nxt.kind == 2) load_relay(i + 1, nxt);
    }
    STAMP(1);
    if (blockIdx.x == 0 && wave == 0) {
        const unsigned* fd = (const unsigned*)(lws + OFF_FD) + lane * 4;
        unsigned* tmo = (unsigned*)(lws + OFF_TMO);
        const u64 t0 = __builtin_amdgcn_s_memrealtime();
        for (;;) {
            u32x4 a; unsigned tm;
            asm volatile("global_load_dwordx4 %0, %2, off sc0 sc1\n\tglobal_load_dword %1, %3, off sc0 sc1\n\ts_waitcnt vmcnt(0)"
                         : "=&v"(a), "=&v"(tm) : "v"(fd), "v"(tmo) : "memory");
            if (__builtin_amdgcn_ballot_w64(min(min(a[0], a[1]), min(a[2], a[3])) >= call) == ~0ull) break;
            if ((unsigned)__builtin_amdgcn_readfirstlane(tm) == call || __builtin_amdgcn_s_memrealtime() - t0 > DEADLINE_TICKS) {
                __hip_atomic_store(tmo, call, __ATOMIC_RELAXED, __HIP_MEMORY_SCOPE_AGENT);
                if (lane == 0) p.out[rank][((rank >> 2) ^ 1) * 2048] = __builtin_nanf("");
                break;
            }
            __builtin_amdgcn_s_sleep(1);
        }
        STAMP(2);
    }
}

__device__ __forceinline__ float silu(float y) {
    const float z = fminf(fmaxf(y, -60.f), 60.f);
    return y * __builtin_amdgcn_rcpf(1.0f + __expf(-z));
}

template <int NB>
__device__ __forceinline__ void tile_role(const Params& p, char* lds, int rb, int cg, unsigned* ctr) {
    constexpr int SLOT = 16384 + NB * 8192;
    constexpr int NS = LDS_RING / SLOT;
    constexpr int MT = 2 * NB;
    enum { C_CALL = 0, C_AVAIL = 1, C_HAND = 2, C_HANDV = 3, C_BAD = 4, C_FULL = 8, C_DONE = 16, C_ORDER = 32 };
    const int lane = threadIdx.x & 63;
    const int wave = __builtin_amdgcn_readfirstlane(threadIdx.x >> 6);
    const int rank = p.rank;
    char* ws = p.ws[rank];
    unsigned* tmo = (unsigned*)(ws + OFF_TMO);
    const ldsw_t ctl = (ldsw_t)(lds + LDS_RING);
    if (threadIdx.x < 32)
        ctl[threadIdx.x] = threadIdx.x == 0 ? __hip_atomic_fetch_add(ctr, 1u, __ATOMIC_RELAXED, __HIP_MEMORY_SCOPE_AGENT) + 1 : 0;
    __syncthreads();
    const unsigned call = lds_get(ctl + C_CALL);
    const unsigned par = call & 1;
    const int hidx = ((rb ^ rank) & 3) * 8 + (cg >> 2);

    if (wave == 4) {
        const unsigned* fx = (const unsigned*)(ws + OFF_FX + par * FL_PAR + (rb * 64 + lane) * 16);
        const unsigned* fw = (const unsigned*)(ws + OFF_FW + par * FL_PAR + (cg * 64 + lane) * 8);
        const u64 t0 = __builtin_amdgcn_s_memrealtime();
        u32x4 a = {0, 0, 0, 0}; u32x2 b = {0, 0}, b2 = {0, 0};
        u64 nx = ~0ull, nw = ~0ull, done = 0;
        int cnt = 0, nb_dbg = 0;
        bool bad = false;
        while (done != ~0ull) {
            unsigned tm; u64 sv;
            if constexpr (NB == 2) {
                asm volatile("s_mov_b64 %4, exec\n\tglobal_load_dword %3, %9, off sc0 sc1\n\t"
                             "s_mov_b64 exec, %7\n\tglobal_load_dwordx4 %0, %5, off sc0 sc1\n\t"
                             "s_mov_b64 exec, %8\n\tglobal_load_dwordx2 %1, %6, off sc0 sc1\n\t"
                             "global_load_dwordx2 %2, %6, off offset:512 sc0 sc1\n\t"
                             "s_mov_b64 exec, %4\n\ts_waitcnt vmcnt(0)"
                             : "+v"(a), "+v"(b), "+v"(b2), "=&v"(tm), "=&s"(sv) : "v"(fx), "v"(fw), "s"(nx), "s"(nw), "v"(tmo) : "memory");
            } else {
                asm volatile("s_mov_b64 %3, exec\n\tglobal_load_dword %2, %8, off sc0 sc1\n\t"
                             "s_mov_b64 exec, %6\n\tglobal_load_dwordx4 %0, %4, off sc0 sc1\n\t"
                             "s_mov_b64 exec, %7\n\tglobal_load_dwordx2 %1, %5, off sc0 sc1\n\t"
                             "s_mov_b64 exec, %3\n\ts_waitcnt vmcnt(0)"
                             : "+v"(a), "+v"(b), "=&v"(tm), "=&s"(sv) : "v"(fx), "v"(fw), "s"(nx), "s"(nw), "v"(tmo) : "memory");
                b2 = b;
            }
            nx = __builtin_amdgcn_ballot_w64(min(min(a[0], a[1]), min(a[2], a[3])) < call);
            nw = __builtin_amdgcn_ballot_w64(min(min(b[0], b[1]), min(b2[0], b2[1])) < call);
            u64 ready = ~(nx | nw | done);
            if (!ready && ((unsigned)__builtin_amdgcn_readfirstlane(tm) == call || __builtin_amdgcn_s_memrealtime() - t0 > DEADLINE_TICKS)) {
                __hip_atomic_store(tmo, call, __ATOMIC_RELAXED, __HIP_MEMORY_SCOPE_AGENT);
                bad = true;
                ctl[C_BAD] = 1;
                ready = ~done;
            }
            if (ready) {
                if ((ready >> lane) & 1)
                    ctl[C_ORDER + cnt + __builtin_amdgcn_mbcnt_hi((unsigned)(ready >> 32), __builtin_amdgcn_mbcnt_lo((unsigned)ready, 0))] = lane;
                cnt += __builtin_popcountll(ready);
                ctl[C_AVAIL] = cnt;
                done |= ready;
                STAMPN(8 + (nb_dbg < 55 ? nb_dbg : 55), cnt);
                ++nb_dbg;
            } else {
                __builtin_amdgcn_s_sleep(2);
            }
        }
        if (NB == 2) {
            const char* ho = ws + OFF_HO + (((cg >> 1) & 1) * 32 + hidx) * 64;
            u32x2 h = {0, 0};
            while (!bad) {
                unsigned tm;
                asm volatile("global_load_dwordx2 %0, %2, off sc0 sc1\n\tglobal_load_dword %1, %3, off sc0 sc1\n\ts_waitcnt vmcnt(0)"
                             : "=&v"(h), "=&v"(tm) : "v"(ho), "v"(tmo) : "memory");
                if ((unsigned)__builtin_amdgcn_readfirstlane(h[1]) >= call) break;
                if ((unsigned)__builtin_amdgcn_readfirstlane(tm) == call || __builtin_amdgcn_s_memrealtime() - t0 > DEADLINE_TICKS) {
                    __hip_atomic_store(tmo, call, __ATOMIC_RELAXED, __HIP_MEMORY_SCOPE_AGENT);
                    bad = true;
                    ctl[C_BAD] = 1;
                }
                __builtin_amdgcn_s_sleep(2);
            }
            ctl[C_HAND] = bad ? now32() : h[0];
            ctl[C_HANDV] = 1;
        }
        STAMP(1);
        return;
    }
    if (wave > 4) {
        const char* xs = ws + par * PAR_BYTES + rb * X_RB + lane * 16;
        const char* wsrc = ws + par * PAR_BYTES + X_BYTES + cg * W_CG + lane * 16;
#pragma unroll 1
        for (int n = wave - 5; n < NSTEP; n += 3) {
            const int slot = n % NS;
            lds_wait(ctl + C_AVAIL, n + 1);
            const unsigned t = lds_get(ctl + C_ORDER + n);
            lds_wait(ctl + C_DONE + slot, 4 * (n / NS));
            char* l = lds + slot * SLOT;
#pragma unroll
            for (int i = 0; i < 4; ++i) glds4k(xs + t * (unsigned)X_STEP + i * 4096, l + i * 4096);
#pragma unroll
            for (int i = 0; i < 2 * NB; ++i) glds4k(wsrc + t * (unsigned)W_STEP + (i >> 1) * W_CG + (i & 1) * 4096, l + 16384 + i * 4096);
            wait_vm0();
            ctl[C_FULL + slot] = n + 1;
        }
        return;
    }

    const int wm = NB == 2 ? wave >> 1 : wave, wn = NB == 2 ? wave & 1 : 0;
    f32x4 acc[MT][4];
#pragma unroll
    for (int m = 0; m < MT; ++m)
#pragma unroll
        for (int n = 0; n < 4; ++n) acc[m][n] = f32x4{0.f, 0.f, 0.f, 0.f};
    int slot = 0;
#pragma unroll 1
    for (int n = 0; n < NSTEP; ++n) {
        lds_wait(ctl + C_FULL + slot, n + 1);
        const char* l = lds + slot * SLOT + lane * 16;
        f16x8 a[2][MT], b[2][4];
#pragma unroll
        for (int kh = 0; kh < 2; ++kh) {
#pragma unroll
            for (int m = 0; m < MT; ++m) a[kh][m] = *(const f16x8*)(l + (wm * MT + m) * 2048 + kh * 1024);
#pragma unroll
            for (int n2 = 0; n2 < 4; ++n2) b[kh][n2] = *(const f16x8*)(l + 16384 + wn * 8192 + kh * 4096 + n2 * 1024);
        }
        asm volatile("" ::: "memory");
        if (lane == 0) __hip_atomic_fetch_add((__attribute__((address_space(3))) unsigned*)(ctl + C_DONE + slot), 1u, __ATOMIC_RELAXED, __HIP_MEMORY_SCOPE_WORKGROUP);
        asm volatile("" ::: "memory");
#pragma unroll
        for (int kh = 0; kh < 2; ++kh)
#pragma unroll
            for (int m = 0; m < MT; ++m)
#pragma unroll
                for (int n2 = 0; n2 < 4; ++n2)
                    acc[m][n2] = __builtin_amdgcn_mfma_f32_16x16x32_f16(a[kh][m], b[kh][n2], acc[m][n2], 0, 0, 0);
        slot = slot == NS - 1 ? 0 : slot + 1;
        if ((n & 3) == 3) STAMP(8 + (n >> 2));
    }
    STAMP(1);

    unsigned when = 0;
    if (NB == 2) {
        lds_wait(ctl + C_HANDV, 1);
        const unsigned h = lds_get(ctl + C_HAND), t = now32() - CREDIT;
        when = (int)(t - h) > 0 ? t : h;
        if (wave == 0 && !(cg & 2)) {
            const u32x2 ho = {when + (16 * TAU >> 5), call};
            if (lane == 0)
                asm volatile("global_store_dwordx2 %0, %1, off sc0 sc1\n\ts_nop 1" :: "v"(ws + OFF_HO + (32 + hidx) * 64), "v"(ho) : "memory");
        }
        when += wave * TAU >> 7;
    }
    const float bad = lds_get(ctl + C_BAD) ? __builtin_nanf("") : 0.f;
    const int grp = rank >> 2;
    const int q = rb < 4 ? rank : ((grp ^ 1) * 4 + (rb - 4));
    const int row0 = (rb < 4 ? rb : (rank & 3)) * 128 + wm * MT * 16 + (lane >> 4) * 4;
    float* o = p.out[q] + (size_t)row0 * NN + grp * 2048 + (cg + wn) * 64 + (lane & 15) * 4;
#pragma unroll
    for (int m = 0; m < MT; ++m)
#pragma unroll
        for (int r = 0; r < 4; ++r) {
            f32x4 v;
#pragma unroll
            for (int i = 0; i < 4; ++i) v[i] = silu(acc[m][i][r]) + bad;
            if (NB == 2) {
                wait_until(when + ((m * 4 + r) * TAU >> 5));
                const unsigned t = now32() - CREDIT - ((m * 4 + r) * TAU >> 5);
                if ((int)(t - when) > 0) when = t;
            }
            store16_sys(o + (size_t)(m * 16 + r) * NN, __builtin_bit_cast(u32x4, v));
            if (NB == 2) wait_vm<ROUT_INFLIGHT>();
        }
    if (NB == 2) {
        wait_vm0();
        if (lane == 0) {
            unsigned* fl = (unsigned*)(p.ws[q] + OFF_FD + (((rank & 3) * 16 + (cg >> 1)) * 4 + wave) * 4);
            __hip_atomic_store(fl, call, __ATOMIC_RELAXED, __HIP_MEMORY_SCOPE_SYSTEM);
        }
    }
    STAMP(2);
}

__global__ void __launch_bounds__(BLOCK, 1) fused_kernel(Params p) {
    extern __shared__ __attribute__((aligned(16))) char lds[];
    STAMP(0);
    unsigned* ctr = (unsigned*)(p.ws[p.rank] + OFF_CT + blockIdx.x * 128);
    const int xcd = blockIdx.x & 7, lb = blockIdx.x >> 3;
    if (lb < 8) {
        push_role(p, lds, lb, xcd, ctr);
    } else if (lb < 24) {
        const int k = lb - 8;
        tile_role<1>(p, lds, k & 3, xcd * 4 + (k >> 2), ctr);
    } else {
        const int k = lb - 24;
        tile_role<2>(p, lds, 4 + (k & 3), xcd * 4 + (k >> 2) * 2, ctr);
    }
}

struct Tab { int n[3]; unsigned char d[3][64]; };
inline Tab make_tab() {
    Tab t{};
    auto add = [&](int cls, int type, int s, int lo2) { t.d[cls][t.n[cls]++] = (unsigned char)(type << 5 | s << 2 | lo2); };
    for (int s = 0; s < 8; ++s) { for (int l = 0; l < 4; ++l) add(0, T_W, s, l); add(0, T_XR, s, 0); add(0, T_XO, s, 0); }
    for (int s = 0; s < 8; ++s) {
        add(1, T_W, s, 0); add(1, T_W, s, 1); add(1, T_XR, s, 0);
        if (s & 1) { add(1, T_W, s >> 1, 2); add(1, T_W, s >> 1, 3); }
        if (s >= 2 && s < 6) { add(1, T_RF, s & 1, (s - 2) & 2); add(1, T_RF, s & 1, ((s - 2) & 2) + 1); }
    }
    for (int s = 4; s < 8; ++s) { add(1, T_W, s, 2); add(1, T_W, s, 3); }
    for (int s = 0; s < 8; ++s) add(1, T_XO, s, 0);
    add(2, T_RS, 0, 0); add(2, T_RS, 1, 0);
    add(2, T_XR, 0, 0); add(2, T_XR, 1, 0); add(2, T_XO, 0, 0); add(2, T_XO, 1, 0);
    for (int s = 2; s < 8; ++s) {
        for (int l = 0; l < (s < 7 ? 4 : 2); ++l) add(2, T_W, s, l);
        add(2, T_XR, s, 0);
        if (s < 7) add(2, T_XO, s, 0);
    }
    add(2, T_W, 7, 2); add(2, T_W, 7, 3); add(2, T_XO, 7, 0);
    return t;
}

}

extern "C" void kernel_launch(int rank, int nranks, ncclComm_t comm,
                              hipStream_t stream,
                              void* const* d_in, const int* in_sizes, int n_in,
                              void* d_out, int out_size,
                              const int* in_dtypes, int out_dtype,
                              void* const* d_out_all,
                              void* const* d_ws_all, size_t ws_bytes) {
    if (nranks != NR || (unsigned)rank >= (unsigned)NR || ws_bytes < WS_NEED || n_in < 2 ||
        in_sizes[0] != MM * KS || in_sizes[1] != KS * NN || out_size != (MM / NR) * NN) {
        fprintf(stderr, "kernel_launch: unexpected shapes\n");
        return;
    }
    HIP_CHECK(hipFuncSetAttribute((const void*)fused_kernel, hipFuncAttributeMaxDynamicSharedMemorySize, LDS_BYTES));
    Params p;
    p.x = (const float*)d_in[0];
    p.w = (const float*)d_in[1];
    for (int i = 0; i < NR; ++i) { p.ws[i] = (char*)d_ws_all[i]; p.out[i] = (float*)d_out_all[i]; }
    p.rank = rank;
    const Tab tab = make_tab();
    for (int i = 0; i < 3; ++i) { p.ntab[i] = tab.n[i]; for (int j = 0; j < 64; ++j) p.tab[i * 64 + j] = tab.d[i][j]; }
    fused_kernel<<<GRID, BLOCK, LDS_BYTES, stream>>>(p);
    HIP_CHECK(hipGetLastError());
}
```

```cpp
#include <hip/hip_runtime.h>
#include <rccl/rccl.h>
#include <cstdio>
#include <cstdint>

#define HIP_CHECK(expr) do { hipError_t _e = (expr); if (_e != hipSuccess) { \
    fprintf(stderr, "HIP error %s at %s:%d\n", hipGetErrorString(_e), __FILE__, __LINE__); \
    return; } } while (0)

typedef unsigned u32x4 __attribute__((ext_vector_type(4)));
typedef unsigned u32x2 __attribute__((ext_vector_type(2)));
typedef unsigned short u16x2 __attribute__((ext_vector_type(2)));
typedef float f32x4 __attribute__((ext_vector_type(4)));
typedef float f32x2 __attribute__((ext_vector_type(2)));
typedef _Float16 f16x8 __attribute__((ext_vector_type(8)));
typedef _Float16 f16x2 __attribute__((ext_vector_type(2)));
typedef unsigned long long u64;

namespace {

constexpr int NR = 8;
constexpr int MM = 4096, NN = 4096;
constexpr int KS = 512;
constexpr int NSTEP = 64;

constexpr unsigned X_STEP = 16384, X_RB = X_STEP * NSTEP, X_BYTES = 8 * X_RB;
constexpr unsigned W_STEP = 8192, W_CG = W_STEP * NSTEP, W_BYTES = 32 * W_CG;
constexpr unsigned PK = 4224;
constexpr unsigned OFF_EX = X_BYTES + W_BYTES;
constexpr unsigned OFF_RX = OFF_EX + 2048 * PK;
constexpr unsigned OFF_OX = OFF_RX + 256 * PK;
constexpr unsigned OFF_FX = OFF_OX + 1024 * 4096;
constexpr unsigned OFF_FW = OFF_FX + 8 * 64 * 16;
constexpr unsigned OFF_FE = OFF_FW + 32 * 64 * 8;
constexpr unsigned OFF_FR = OFF_FE + 8 * 512;
constexpr unsigned OFF_FD = OFF_FR + 4096;
constexpr unsigned PAR_BYTES = OFF_FD + 1024;
constexpr unsigned OFF_TMO = 2 * PAR_BYTES;
constexpr unsigned OFF_TK = OFF_TMO + 1024;
constexpr unsigned OFF_CT = OFF_TK + 1024;
constexpr unsigned OFF_DBG = OFF_CT + 2048 * 16;
constexpr size_t WS_NEED = OFF_DBG + 256 * 8 * 64 * 8;

constexpr int GRID = 256, BLOCK = 512;
#define TAU 1740
#define TAUP 1900
#define CREDIT 0
#define NPUSH 2
#define NREL 7
#define LATE 0
constexpr int NWP = 64;
constexpr unsigned long long DEADLINE_TICKS = 50000000ull;

constexpr int LDS_RING = 6 * 24576;
constexpr int LDS_PACK = LDS_RING + 512;
constexpr int LDS_BYTES = LDS_PACK + 8 * 640;
typedef volatile __attribute__((address_space(3))) unsigned* ldsw_t;
typedef __attribute__((address_space(3))) void* ldsp_t;
typedef const __attribute__((address_space(1))) void* glbp_t;
enum { C_TZ = 0, C_AVAIL = 1, C_CLK = 2, C_PCLK = 3, C_BAD = 4, C_PACKED = 5, C_FULL = 8, C_DONE = 16, C_ORDER = 32 };

struct Params {
    const float* x;
    const float* w;
    char* lws;
    int rank, pad;
    float* out;
    char* ws[NR];
    int ntab[2];
    unsigned char tab[2 * 64];
};

__device__ __forceinline__ void store16_sys(void* p, u32x4 v) {
    asm volatile("global_store_dwordx4 %0, %1, off sc0 sc1\n\ts_nop 1" :: "v"(p), "v"(v) : "memory");
}
__device__ __forceinline__ void store4k_sys(char* base, unsigned off, u32x4 a, u32x4 b, u32x4 c, u32x4 d) {
    asm volatile("global_store_dwordx4 %0, %1, %5 sc0 sc1\n\t"
                 "global_store_dwordx4 %0, %2, %5 offset:1024 sc0 sc1\n\t"
                 "global_store_dwordx4 %0, %3, %5 offset:2048 sc0 sc1\n\t"
                 "global_store_dwordx4 %0, %4, %5 offset:3072 sc0 sc1\n\ts_nop 1"
                 :: "v"(off), "v"(a), "v"(b), "v"(c), "v"(d), "s"(base) : "memory");
}
__device__ __forceinline__ void store_packed(char* base, unsigned off, u32x4 a, u32x4 b, u32x4 c, u32x4 d) {
    u64 sv;
    asm volatile("global_store_dwordx4 %1, %2, %6 sc0 sc1\n\t"
                 "global_store_dwordx4 %1, %3, %6 offset:1024 sc0 sc1\n\t"
                 "global_store_dwordx4 %1, %4, %6 offset:2048 sc0 sc1\n\t"
                 "s_mov_b64 %0, exec\n\ts_mov_b32 exec_hi, 0xff\n\t"
                 "global_store_dwordx4 %1, %5, %6 offset:3072 sc0 sc1\n\ts_mov_b64 exec, %0\n\ts_nop 1"
                 : "=&s"(sv) : "v"(off), "v"(a), "v"(b), "v"(c), "v"(d), "s"(base) : "memory");
}
__device__ __forceinline__ void flag_sys(char* base, unsigned off, unsigned v) {
    u64 sv;
    asm volatile("s_mov_b64 %0, exec\n\ts_mov_b64 exec, 1\n\tglobal_store_dword %1, %2, %3 sc0 sc1\n\ts_mov_b64 exec, %0"
                 : "=&s"(sv) : "v"(off), "v"(v), "s"(base) : "memory");
}
__device__ __forceinline__ void wait_vm0() { asm volatile("s_waitcnt vmcnt(0)" ::: "memory"); }

template <class T> __device__ __forceinline__ T* uniform_ptr(T* p) {
    const u64 v = (u64)p;
    return (T*)(((u64)(unsigned)__builtin_amdgcn_readfirstlane((unsigned)(v >> 32)) << 32) | (u64)(unsigned)__builtin_amdgcn_readfirstlane((unsigned)v));
}

__device__ __forceinline__ unsigned now32() { return (unsigned)__builtin_amdgcn_s_memrealtime(); }
__device__ __forceinline__ void wait_until(unsigned t) {
    while ((int)(now32() - t) < 0) __builtin_amdgcn_s_sleep(1);
}

__device__ __forceinline__ unsigned pack2(float a, float b) {
    f32x2 v = {a, b};
    return __builtin_bit_cast(unsigned, __builtin_convertvector(v, f16x2));
}
__device__ __forceinline__ unsigned bfi(unsigned m, unsigned a, unsigned b) { return (a & m) | (b & ~m); }

#define STAMPN(idx, n) do {} while (0)
#define STAMP(idx) do {} while (0)

__device__ __forceinline__ unsigned lds_get(ldsw_t w) { return __builtin_amdgcn_readfirstlane(*w); }
__device__ __forceinline__ void lds_wait(ldsw_t w, unsigned v) {
    while (lds_get(w) < v) __builtin_amdgcn_s_sleep(1);
    asm volatile("" ::: "memory");
}
__device__ __forceinline__ void glds4k(const char* g, char* l) {
    __builtin_amdgcn_global_load_lds((glbp_t)g, (ldsp_t)l, 16, 0, 17);
    __builtin_amdgcn_global_load_lds((glbp_t)g, (ldsp_t)l, 16, 1024, 17);
    __builtin_amdgcn_global_load_lds((glbp_t)g, (ldsp_t)l, 16, 2048, 17);
    __builtin_amdgcn_global_load_lds((glbp_t)g, (ldsp_t)l, 16, 3072, 17);
}
__device__ __forceinline__ unsigned lds_add(ldsw_t w, unsigned v) {
    unsigned r; u64 sv;
    asm volatile("s_mov_b64 %1, exec\n\ts_mov_b64 exec, 1\n\tds_add_rtn_u32 %0, %2, %3\n\ts_mov_b64 exec, %1\n\ts_waitcnt lgkmcnt(0)"
                 : "=&v"(r), "=&s"(sv) : "v"(w), "v"(v) : "memory");
    return __builtin_amdgcn_readfirstlane(r);
}
__device__ __forceinline__ void pace(ldsw_t clk, unsigned tz, unsigned d) {
    unsigned slot; u64 sv;
    asm volatile("s_mov_b64 %1, exec\n\ts_mov_b64 exec, 1\n\tds_max_u32 %2, %3\n\tds_add_rtn_u32 %0, %2, %4\n\t"
                 "s_mov_b64 exec, %1\n\ts_waitcnt lgkmcnt(0)"
                 : "=&v"(slot), "=&s"(sv) : "v"(clk), "v"((now32() - tz) << 5), "v"(d) : "memory");
    wait_until(tz - CREDIT + ((unsigned)__builtin_amdgcn_readfirstlane(slot) >> 5));
}
__device__ __forceinline__ int phase_cg(int ph, int xx, int c1) { return 8 * xx + 4 * (ph & 1) + (ph & 2) + c1; }

__device__ __forceinline__ unsigned pack_role(const Params& p, char* lds) {
    const int lane = threadIdx.x & 63;
    const int wave = __builtin_amdgcn_readfirstlane(threadIdx.x >> 6);
    const int b = blockIdx.x;
    int rank = p.rank;
    const int c = lane & 15, kq = lane >> 4;
    const ldsw_t ctl = (ldsw_t)(lds + LDS_RING);
    const bool isx = b < 128;
    const float* px = p.x; const float* pw = p.w; char* lws = p.lws;
    asm volatile("" :: "s"(px), "s"(pw), "s"(lws), "s"(rank));
    const int R = b & 31, sp = b >> 5, sub = wave & 3, q = R >> 2;
    const int hh = b & 1, ph = (b >> 1) & 3, g = (b >> 3) & 1, kh = wave & 1;
    const int cg = phase_cg(ph, 2 * hh + (wave >> 2), (wave >> 1) & 1);
    const int s = isx ? 2 * sp + (wave >> 2) : (b >> 4) & 7, t = rank * 8 + s;
    const char* base = (const char*)(isx ? px + ((R * 128 + sub * 32) * KS + s * 64) : pw + ((s * 64 + kh * 32) * NN + g * 2048 + cg * 64));
    const unsigned voff = isx ? (c * KS + 4 * kq) * 4 : (4 * kq * NN + 4 * c) * 4;
    const unsigned sa = isx ? 64 : NN * 4, sb = isx ? 16 * KS * 4 : 16 * NN * 4;
    if (LATE && b >= 16 * NPUSH) __builtin_amdgcn_s_sleep(LATE);
    unsigned* const ctr = (unsigned*)(lws + OFF_CT) + (b * 8 + wave) * 4;
    const unsigned cv = __hip_atomic_load(ctr, __ATOMIC_RELAXED, __HIP_MEMORY_SCOPE_SYSTEM);
    f32x4 v[8];
#pragma unroll
    for (int jj = 0; jj < 8; ++jj) v[jj] = *(const f32x4*)(base + (voff + (jj & 3) * sa + (jj >> 2) * sb));
    asm volatile("" ::: "memory");
    unsigned eu, d1, f1, fe;
    int q1, fq; bool on1;
    if (isx) {
        eu = (R * 4 + sub) * 8 + s;
        q1 = ((q & 4) ^ 4) + (R & 3);
        on1 = q1 == rank || s < NPUSH;
        d1 = (4 + (q & 3)) * X_RB + t * X_STEP + sub * 4096; f1 = OFF_FX + ((4 + (q & 3)) * 64 + t) * 16 + sub * 4;
        fq = lane & 1 ? q1 : q;
        fe = lane & 1 ? 80 + (q & 3) * 4 + sp : 64 + (R & 3) * 4 + sp;
    } else {
        eu = 1024 + (g * 32 + cg) * 16 + s * 2 + kh;
        q1 = rank;
        on1 = g == rank >> 2;
        d1 = X_BYTES + cg * W_CG + t * W_STEP + kh * 4096; f1 = OFF_FW + (cg * 64 + t) * 8 + kh * 4;
        fq = 4 * g + (lane & 3);
        fe = (s * 4 + ph) * 2 + hh;
    }
    const bool on2 = isx && q == rank;
    const unsigned d2 = (R & 3) * X_RB + t * X_STEP + sub * 4096, f2 = OFF_FX + ((R & 3) * 64 + t) * 16 + sub * 4;
    char* const fws = p.ws[fq];
    char* const ws1 = uniform_ptr(p.ws[q1]);
    char* const sc = lds + LDS_PACK + wave * 640;
    if (threadIdx.x < 32) ctl[threadIdx.x] = threadIdx.x == C_TZ ? now32() : 0;
    if (lane < 32) *(unsigned*)(sc + 512 + lane * 4) = 0;
    asm volatile("s_waitcnt lgkmcnt(0)\n\ts_barrier" ::: "memory");

    unsigned h[16];
    if (isx) {
#pragma unroll
        for (int i = 0; i < 8; ++i) { h[2 * i] = pack2(v[i][0], v[i][1]); h[2 * i + 1] = pack2(v[i][2], v[i][3]); }
    } else {
#pragma unroll
        for (int i = 0; i < 16; ++i) h[i] = pack2(v[2 * (i & 3)][i >> 2], v[2 * (i & 3) + 1][i >> 2]);
    }
    const unsigned call = __builtin_amdgcn_readfirstlane(cv) + 1;
    const unsigned po = (call & 1) * PAR_BYTES;
    STAMP(3);
    if (on1) {
        if (q1 != rank) pace(ctl + C_PCLK, lds_get(ctl + C_TZ), (TAUP * NPUSH / 4) << (2 * (b >> 5) + 2 > NPUSH));
        store4k_sys(ws1, po + d1 + lane * 16, u32x4{h[0], h[1], h[2], h[3]}, u32x4{h[4], h[5], h[6], h[7]},
                    u32x4{h[8], h[9], h[10], h[11]}, u32x4{h[12], h[13], h[14], h[15]});
    }
    if (on2)
        store4k_sys(lws, po + d2 + lane * 16, u32x4{h[0], h[1], h[2], h[3]}, u32x4{h[4], h[5], h[6], h[7]},
                    u32x4{h[8], h[9], h[10], h[11]}, u32x4{h[12], h[13], h[14], h[15]});
    STAMP(4);
    unsigned es = ((h[0] >> 10) & 0x001f001fu) + ((h[5] >> 10) & 0x001f001fu) + ((h[10] >> 10) & 0x001f001fu) + ((h[15] >> 10) & 0x001f001fu);
    es = (es & 0xffffu) + (es >> 16);
    es += __builtin_amdgcn_update_dpp(0u, es, 0x111, 0xf, 0xf, true);
    es += __builtin_amdgcn_update_dpp(0u, es, 0x112, 0xf, 0xf, true);
    es += __builtin_amdgcn_update_dpp(0u, es, 0x114, 0xf, 0xf, true);
    es += __builtin_amdgcn_update_dpp(0u, es, 0x118, 0xf, 0xf, true);
    const int sum = __builtin_amdgcn_readlane(es, 15) + __builtin_amdgcn_readlane(es, 31) + __builtin_amdgcn_readlane(es, 47) + __builtin_amdgcn_readlane(es, 63);
    const unsigned eb = min(max((sum - 2135) >> 9, 0), 24);
    unsigned pl[2] = {0, 0};
#pragma unroll
    for (int j = 0; j < 16; ++j) {
        h[j] = __builtin_bit_cast(unsigned, __builtin_bit_cast(u16x2, h[j]) - __builtin_bit_cast(u16x2, eb * 0x04000400u));
        pl[j >> 3] |= ((j & 7) == 7 ? h[j] << 1 : h[j] >> (13 - 2 * (j & 7))) & (0x00030003u << 2 * (j & 7));
    }
#pragma unroll
    for (int k = 0; k < 2; ++k) {
        const unsigned d = bfi(0x20002000u, h[14 + k] >> 2, h[14 + k]);
#pragma unroll
        for (int j = 0; j < 7; ++j) h[7 * k + j] = bfi(0x60006000u, d << (13 - 2 * j), h[7 * k + j]);
    }
    *(u32x2*)(sc + lane * 8) = u32x2{h[12], h[13]};
    int cnt = 0;
#pragma unroll
    for (int k = 0; k < 2; ++k) {
        unsigned m = pl[k];
        u64 any;
        while ((any = __builtin_amdgcn_ballot_w64(m != 0)) != 0 && cnt <= 63) {
            const unsigned bit = __builtin_ctz(m | 0x80000000u) & 30, val = (m >> bit) & 3;
            const int pos = cnt + __builtin_amdgcn_mbcnt_hi((unsigned)(any >> 32), __builtin_amdgcn_mbcnt_lo((unsigned)any, 0));
            if (m != 0 && pos < 63) *(unsigned short*)(sc + 512 + pos * 2) = (unsigned short)((lane * 2 + k) << 6 | bit << 1 | val);
            m &= ~(3u << bit);
            cnt += __builtin_popcountll(any);
        }
    }
    if (lane == 0) *(unsigned short*)(sc + 512 + 126) = (unsigned short)(eb << 2 | (cnt > 63 ? 128 : 0));
    asm volatile("" ::: "memory");
    const u32x4 tail = *(const u32x4*)(sc + (lane < 40 ? lane : 0) * 16);
    store_packed(lws, po + OFF_EX + eu * PK + lane * 16, u32x4{h[0], h[1], h[2], h[3]}, u32x4{h[4], h[5], h[6], h[7]},
                 u32x4{h[8], h[9], h[10], h[11]}, tail);
    if (cnt > 63)
        asm volatile("global_store_dwordx2 %0, %1, %2 offset:3712 sc0 sc1\n\ts_nop 1" :: "v"(po + OFF_EX + eu * PK + lane * 8), "v"(u32x2{pl[0], pl[1]}), "s"(lws) : "memory");
    STAMP(5);
    wait_vm0();
    if (on1) flag_sys(ws1, po + f1, call);
    if (on2) flag_sys(lws, po + f2, call);
    flag_sys(lws, OFF_CT + (b * 8 + wave) * 16, call);
    if (lds_add(ctl + C_PACKED, 1) == 7 && lane < (isx ? 2 : 4))
        asm volatile("global_store_dword %0, %1, off sc0 sc1" :: "v"(fws + po + OFF_FE + rank * 512 + fe * 4), "v"(call) : "memory");
    STAMP(6);
    return call;
}

enum { T_W = 0, T_XR = 1, T_XO = 2, T_RS = 3, T_RF = 4, T_OUT = 5 };
enum { K_IN = 0, K_RS = 1, K_OUT = 2, K_NONE = 3 };
struct Task { int kind; unsigned src, wf, dst, fl, rel; };

__device__ __forceinline__ void pull_role(const Params& p, char* lds, int e, int xcd, unsigned call) {
    const int lane = threadIdx.x & 63;
    const int wave = __builtin_amdgcn_readfirstlane(threadIdx.x >> 6);
    const int jw = wave * 8 + xcd;
    char* const lb = lds + wave * 8192;
    const ldsw_t ctl = (ldsw_t)(lds + LDS_RING);
    const int rank = p.rank, q = rank ^ e;
    const int cls = e >> 2;
    int desc = p.tab[cls * 64 + lane];
    const int ntask = p.ntab[cls] * 16;
    const unsigned po = (call & 1) * PAR_BYTES;
    const char* const sws = p.ws[q] + po;
    char* const lws = p.lws;
    char* const pws = p.ws[rank ^ (lane % 3 + 1)] + po + OFF_FR + (rank & 3) * 1024;
    unsigned* const tmo = (unsigned*)(lws + OFF_TMO);
    const unsigned tz = lds_get(ctl + C_TZ);

    auto decode = [&](int u, Task& tk) {
        const int d = __builtin_amdgcn_readlane(desc, u >> 4), j = u & 15;
        const int type = d >> 5, a = d & 31;
        tk.kind = K_IN; tk.rel = 0;
        if (type == T_XR || type == T_XO) {
            const int rbl = j >> 2, sub = j & 3, t = q * 8 + a;
            const int row = type == T_XO ? rank * 16 + j : (((rank & 4) ^ 4) + rbl) * 16 + (rank & 3) * 4 + sub;
            const int rb = type == T_XO ? rbl : 4 + rbl;
            tk.src = OFF_EX + (row * 8 + a) * PK;
            tk.wf = OFF_FE + q * 512 + (64 + rb * 4 + (a >> 1)) * 4;
            tk.dst = rb * X_RB + t * X_STEP + sub * 4096;
            tk.fl = OFF_FX + (rb * 64 + t) * 16 + sub * 4;
        } else if (type == T_OUT) {
            const int pair = phase_cg(a >> 2, a & 3, 0) >> 1;
            tk.kind = K_OUT;
            tk.src = OFF_OX + (((rank & 3) * 16 + pair) * 16 + j) * 4096;
            tk.wf = OFF_FD + (((q & 3) * 16 + pair) * 4 + (j >> 2)) * 4;
            tk.dst = (((q & 3) * 128 + (j >> 3) * 64 + (j & 3) * 16) * NN + (q >> 2) * 2048 + pair * 128 + ((j >> 2) & 1) * 64) * 4;
            tk.fl = 0;
        } else {
            const int r = type == T_RS ? 4 * a + (j >> 2) : a, s = r >> 2, ph = r & 3, kh = j & 1;
            const int xx = type == T_W ? j >> 2 : (type == T_RS ? rank & 3 : q & 3);
            const int cg = phase_cg(ph, xx, (j >> 1) & 1);
            const int t = (type == T_RF ? ((rank & 4) ^ 4) + (j >> 2) : q) * 8 + s;
            tk.rel = (((type == T_RF ? j >> 2 : q & 3) * 16 + r) * 4 + (j & 3));
            tk.src = type == T_RF ? OFF_RX + tk.rel * PK : OFF_EX + (1024 + ((rank >> 2) * 32 + cg) * 16 + s * 2 + kh) * PK;
            tk.wf = type == T_RF ? OFF_FR + (q & 3) * 1024 + tk.rel * 4 : OFF_FE + q * 512 + (r * 2 + (xx >> 1)) * 4;
            tk.dst = X_BYTES + cg * W_CG + t * W_STEP + kh * 4096;
            tk.fl = OFF_FW + (cg * 64 + t) * 8 + kh * 4;
            if (type == T_RS) tk.kind = r < NREL ? K_RS : K_NONE;
        }
    };

    asm volatile("" : "+v"(desc) :: "memory");
    Task cur, nxt;
    decode(jw, cur);
    nxt = cur;
    int un = NWP + jw;
    unsigned seen = 0;
#pragma unroll 1
    for (int i = 0;; ++i) {
        bool ok = cur.kind != K_NONE;
        if (ok && seen < call) {
            const unsigned* fl = (const unsigned*)(lws + po + cur.wf);
            const u64 t0 = __builtin_amdgcn_s_memrealtime();
            while ((unsigned)__builtin_amdgcn_readfirstlane(__hip_atomic_load(fl, __ATOMIC_RELAXED, __HIP_MEMORY_SCOPE_SYSTEM)) < call) {
                if ((unsigned)__builtin_amdgcn_readfirstlane(__hip_atomic_load(tmo, __ATOMIC_RELAXED, __HIP_MEMORY_SCOPE_AGENT)) == call ||
                    __builtin_amdgcn_s_memrealtime() - t0 > DEADLINE_TICKS) {
                    __hip_atomic_store(tmo, call, __ATOMIC_RELAXED, __HIP_MEMORY_SCOPE_AGENT);
                    ok = false;
                    break;
                }
                __builtin_amdgcn_s_sleep(2);
            }
        }
        asm volatile("" ::: "memory");
        const bool have = un < ntask;
        if (have) decode(un, nxt);
        if (ok) {
            pace(ctl + C_CLK, tz, cur.kind == K_OUT ? TAU : TAU * 29 / 32);
            const char* g = sws + cur.src + lane * 16;
            __builtin_amdgcn_global_load_lds((glbp_t)g, (ldsp_t)lb, 16, 0, 17);
            __builtin_amdgcn_global_load_lds((glbp_t)g, (ldsp_t)lb, 16, 1024, 17);
            __builtin_amdgcn_global_load_lds((glbp_t)g, (ldsp_t)lb, 16, 2048, 17);
            if (lane < (cur.kind == K_OUT ? 64 : 40)) __builtin_amdgcn_global_load_lds((glbp_t)g, (ldsp_t)lb, 16, 3072, 17);
        }
        STAMPN(8 + min(2 * i, 54), un);
        unsigned cv, fv; u64 sv;
        asm volatile("s_mov_b64 %2, exec\n\ts_mov_b64 exec, 1\n\tglobal_atomic_add %0, %3, %4, %6 sc0\n\t"
                     "global_load_dword %1, %5, %6 sc0 sc1\n\ts_mov_b64 exec, %2\n\ts_waitcnt vmcnt(0)"
                     : "=&v"(cv), "=&v"(fv), "=&s"(sv) : "v"(OFF_TK + e * 128), "v"(have ? 1u : 0u), "v"(po + nxt.wf), "s"(lws) : "memory");
        STAMPN(9 + min(2 * i, 54), cur.kind);
        seen = __builtin_amdgcn_readfirstlane(fv);
        const unsigned tv = __builtin_amdgcn_readfirstlane(cv);
        if (cur.kind == K_OUT) {
            char* o = (char*)p.out + cur.dst + ((lane >> 4) * 4 * NN + (lane & 15) * 4) * 4;
#pragma unroll
            for (int r = 0; r < 4; ++r) {
                u32x4 y = *(const u32x4*)(lb + r * 1024 + lane * 16);
                if (!ok) y = u32x4{0x7fc00000u, 0x7fc00000u, 0x7fc00000u, 0x7fc00000u};
                store16_sys(o + r * NN * 4, y);
            }
        } else if (ok) {
            const unsigned ent = *(const unsigned short*)(lb + 3584 + lane * 2);
            u32x4 m[3];
#pragma unroll
            for (int k = 0; k < 3; ++k) m[k] = *(const u32x4*)(lb + k * 1024 + lane * 16);
            const u32x2 m2 = *(const u32x2*)(lb + 3072 + lane * 8);
            const unsigned hdr = __builtin_amdgcn_readlane(ent, 63);
            if (hdr & 128) {
                pace(ctl + C_CLK, tz, TAU / 8);
                if (lane < 32) __builtin_amdgcn_global_load_lds((glbp_t)(sws + cur.src + lane * 16), (ldsp_t)lb, 16, 3712, 17);
                wait_vm0();
            } else {
                *(u32x2*)(lb + 3712 + lane * 8) = u32x2{0, 0};
            }
            asm volatile("" ::: "memory");
            __hip_atomic_fetch_or((__attribute__((address_space(3))) unsigned*)(lb + 3712) + (ent >> 6), (ent & 3) << ((ent >> 1) & 30),
                                  __ATOMIC_RELAXED, __HIP_MEMORY_SCOPE_WORKGROUP);
            asm volatile("" ::: "memory");
            const u32x2 pl = *(const u32x2*)(lb + 3712 + lane * 8);
            if (cur.kind == K_RS) {
                store_packed(lws, po + OFF_RX + cur.rel * PK + lane * 16, m[0], m[1], m[2], *(const u32x4*)(lb + 3072 + lane * 16));
                if (hdr & 128)
                    asm volatile("global_store_dwordx2 %0, %1, %2 offset:3712 sc0 sc1\n\ts_nop 1" :: "v"(po + OFF_RX + cur.rel * PK + lane * 8), "v"(pl), "s"(lws) : "memory");
            }
            unsigned h[16];
#pragma unroll
            for (int j = 0; j < 12; ++j) h[j] = m[j >> 2][j & 3];
            h[12] = m2[0]; h[13] = m2[1];
#pragma unroll
            for (int k = 0; k < 2; ++k) {
                unsigned d = 0;
#pragma unroll
                for (int j = 0; j < 7; ++j) d |= (h[7 * k + j] >> (13 - 2 * j)) & (0x00030003u << 2 * j);
                h[14 + k] = bfi(0x80008000u, d << 2, d);
            }
#pragma unroll
            for (int j = 0; j < 16; ++j) {
                const unsigned t = bfi(0x60006000u, (j & 7) == 7 ? pl[j >> 3] >> 1 : pl[j >> 3] << (13 - 2 * (j & 7)), h[j]);
                h[j] = __builtin_bit_cast(unsigned, __builtin_bit_cast(u16x2, t) + __builtin_bit_cast(u16x2, ((hdr >> 2) & 31) * 0x04000400u));
            }
            store4k_sys(lws, po + cur.dst + lane * 16, u32x4{h[0], h[1], h[2], h[3]}, u32x4{h[4], h[5], h[6], h[7]},
                        u32x4{h[8], h[9], h[10], h[11]}, u32x4{h[12], h[13], h[14], h[15]});
            wait_vm0();
            flag_sys(lws, po + cur.fl, call);
            if (cur.kind == K_RS && lane < 3)
                asm volatile("global_store_dword %0, %1, off sc0 sc1" :: "v"(pws + cur.rel * 4), "v"(call) : "memory");
        }
        if (!have) break;
        cur = nxt;
        un = (int)min(tv - (call - 1) * (unsigned)(ntask - NWP) + 2 * NWP, (unsigned)ntask);
    }
    STAMP(1);
}

__device__ __forceinline__ float silu(float y) {
    const float z = fminf(fmaxf(y, -60.f), 60.f);
    return y * __builtin_amdgcn_rcpf(1.0f + __expf(-z));
}

template <int NB>
__device__ __forceinline__ void tile_role(const Params& p, char* lds, int rb, int cg, unsigned call) {
    constexpr int SLOT = 16384 + NB * 8192;
    constexpr int NS = LDS_RING / SLOT;
    constexpr int MT = 2 * NB;
    const int lane = threadIdx.x & 63;
    const int wave = __builtin_amdgcn_readfirstlane(threadIdx.x >> 6);
    const int rank = p.rank;
    const unsigned po = (call & 1) * PAR_BYTES;
    char* ws = p.lws + po;
    unsigned* tmo = (unsigned*)(p.lws + OFF_TMO);
    const ldsw_t ctl = (ldsw_t)(lds + LDS_RING);

    if (wave == 4) {
        const unsigned* fx = (const unsigned*)(ws + OFF_FX + (rb * 64 + lane) * 16);
        const unsigned* fw = (const unsigned*)(ws + OFF_FW + (cg * 64 + lane) * 8);
        const u64 t0 = __builtin_amdgcn_s_memrealtime();
        u32x4 a = {0, 0, 0, 0}; u32x2 b = {0, 0}, b2 = {0, 0};
        u64 nx = ~0ull, nw = ~0ull, done = 0;
        int cnt = 0, nb_dbg = 0;
        while (done != ~0ull) {
            unsigned tm; u64 sv;
            if constexpr (NB == 2) {
                asm volatile("s_mov_b64 %4, exec\n\tglobal_load_dword %3, %9, off sc0 sc1\n\t"
                             "s_mov_b64 exec, %7\n\tglobal_load_dwordx4 %0, %5, off sc0 sc1\n\t"
                             "s_mov_b64 exec, %8\n\tglobal_load_dwordx2 %1, %6, off sc0 sc1\n\t"
                             "global_load_dwordx2 %2, %6, off offset:512 sc0 sc1\n\t"
                             "s_mov_b64 exec, %4\n\ts_waitcnt vmcnt(0)"
                             : "+v"(a), "+v"(b), "+v"(b2), "=&v"(tm), "=&s"(sv) : "v"(fx), "v"(fw), "s"(nx), "s"(nw), "v"(tmo) : "memory");
            } else {
                asm volatile("s_mov_b64 %3, exec\n\tglobal_load_dword %2, %8, off sc0 sc1\n\t"
                             "s_mov_b64 exec, %6\n\tglobal_load_dwordx4 %0, %4, off sc0 sc1\n\t"
                             "s_mov_b64 exec, %7\n\tglobal_load_dwordx2 %1, %5, off sc0 sc1\n\t"
                             "s_mov_b64 exec, %3\n\ts_waitcnt vmcnt(0)"
                             : "+v"(a), "+v"(b), "=&v"(tm), "=&s"(sv) : "v"(fx), "v"(fw), "s"(nx), "s"(nw), "v"(tmo) : "memory");
                b2 = b;
            }
            nx = __builtin_amdgcn_ballot_w64(min(min(a[0], a[1]), min(a[2], a[3])) < call);
            nw = __builtin_amdgcn_ballot_w64(min(min(b[0], b[1]), min(b2[0], b2[1])) < call);
            u64 ready = ~(nx | nw | done);
            if (!ready && ((unsigned)__builtin_amdgcn_readfirstlane(tm) == call || __builtin_amdgcn_s_memrealtime() - t0 > DEADLINE_TICKS)) {
                __hip_atomic_store(tmo, call, __ATOMIC_RELAXED, __HIP_MEMORY_SCOPE_AGENT);
                ctl[C_BAD] = 1;
                ready = ~done;
            }
            if (ready) {
                if ((ready >> lane) & 1)
                    ctl[C_ORDER + cnt + __builtin_amdgcn_mbcnt_hi((unsigned)(ready >> 32), __builtin_amdgcn_mbcnt_lo((unsigned)ready, 0))] = lane;
                cnt += __builtin_popcountll(ready);
                ctl[C_AVAIL] = cnt;
                done |= ready;
                STAMPN(8 + (nb_dbg < 55 ? nb_dbg : 55), cnt);
                ++nb_dbg;
            } else {
                __builtin_amdgcn_s_sleep(2);
            }
        }
        STAMP(1);
        return;
    }
    if (wave > 4) {
        const char* xs = ws + rb * X_RB + lane * 16;
        const char* wsrc = ws + X_BYTES + cg * W_CG + lane * 16;
#pragma unroll 1
        for (int n = wave - 5; n < NSTEP; n += 3) {
            const int slot = n % NS;
            lds_wait(ctl + C_AVAIL, n + 1);
            const unsigned t = lds_get(ctl + C_ORDER + n);
            lds_wait(ctl + C_DONE + slot, 4 * (n / NS));
            char* l = lds + slot * SLOT;
#pragma unroll
            for (int i = 0; i < 4; ++i) glds4k(xs + t * X_STEP + i * 4096, l + i * 4096);
#pragma unroll
            for (int i = 0; i < 2 * NB; ++i) glds4k(wsrc + t * W_STEP + (i >> 1) * W_CG + (i & 1) * 4096, l + 16384 + i * 4096);
            wait_vm0();
            ctl[C_FULL + slot] = n + 1;
        }
        return;
    }

    const int wm = NB == 2 ? wave >> 1 : wave, wn = NB == 2 ? wave & 1 : 0;
    f32x4 acc[MT][4];
#pragma unroll
    for (int m = 0; m < MT; ++m)
#pragma unroll
        for (int n = 0; n < 4; ++n) acc[m][n] = f32x4{0.f, 0.f, 0.f, 0.f};
    int slot = 0;
#pragma unroll 1
    for (int n = 0; n < NSTEP; ++n) {
        lds_wait(ctl + C_FULL + slot, n + 1);
        const char* l = lds + slot * SLOT + lane * 16;
        f16x8 a[2][MT], b[2][4];
#pragma unroll
        for (int kh = 0; kh < 2; ++kh) {
#pragma unroll
            for (int m = 0; m < MT; ++m) a[kh][m] = *(const f16x8*)(l + (wm * MT + m) * 2048 + kh * 1024);
#pragma unroll
            for (int n2 = 0; n2 < 4; ++n2) b[kh][n2] = *(const f16x8*)(l + 16384 + wn * 8192 + kh * 4096 + n2 * 1024);
        }
        asm volatile("" ::: "memory");
        if (lane == 0) __hip_atomic_fetch_add((__attribute__((address_space(3))) unsigned*)(ctl + C_DONE + slot), 1u, __ATOMIC_RELAXED, __HIP_MEMORY_SCOPE_WORKGROUP);
        asm volatile("" ::: "memory");
#pragma unroll
        for (int kh = 0; kh < 2; ++kh)
#pragma unroll
            for (int m = 0; m < MT; ++m)
#pragma unroll
                for (int n2 = 0; n2 < 4; ++n2)
                    acc[m][n2] = __builtin_amdgcn_mfma_f32_16x16x32_f16(a[kh][m], b[kh][n2], acc[m][n2], 0, 0, 0);
        slot = slot == NS - 1 ? 0 : slot + 1;
        if ((n & 3) == 3) STAMP(8 + (n >> 2));
    }
    STAMP(1);

    const float bad = lds_get(ctl + C_BAD) ? __builtin_nanf("") : 0.f;
    char* o = NB == 1 ? (char*)(p.out + (size_t)(rb * 128 + wave * 32 + (lane >> 4) * 4) * NN + (rank >> 2) * 2048 + cg * 64 + (lane & 15) * 4)
                      : ws + OFF_OX + (((rb - 4) * 16 + (cg >> 1)) * 4 + wave) * 16384 + lane * 16;
#pragma unroll
    for (int m = 0; m < MT; ++m)
#pragma unroll
        for (int r = 0; r < 4; ++r) {
            f32x4 v;
#pragma unroll
            for (int i = 0; i < 4; ++i) v[i] = silu(acc[m][i][r]) + bad;
            store16_sys(o + (NB == 1 ? (m * 16 + r) * NN * 4 : (m * 4 + r) * 1024), __builtin_bit_cast(u32x4, v));
        }
    if (NB == 2) {
        wait_vm0();
        flag_sys(uniform_ptr(p.ws[((rank & 4) ^ 4) + rb - 4]), po + OFF_FD + (((rank & 3) * 16 + (cg >> 1)) * 4 + wave) * 4, call);
    }
    STAMP(2);
}

__global__ void __launch_bounds__(BLOCK, 1) fused_kernel(Params p) {
    extern __shared__ __attribute__((aligned(16))) char lds[];
    STAMP(0);
    const unsigned call = pack_role(p, lds);
    const int xcd = blockIdx.x & 7, lb = blockIdx.x >> 3;
    if (lb < 8) {
        tile_role<2>(p, lds, 4 + (lb & 3), xcd * 4 + (lb >> 2) * 2, call);
    } else if (lb < 24) {
        const int k = lb - 8;
        tile_role<1>(p, lds, k & 3, xcd * 4 + (k >> 2), call);
    } else if (lb > 24) {
        pull_role(p, lds, lb - 24, xcd, call);
    }
}

struct Tab { int n[2]; unsigned char d[2][64]; };
inline Tab make_tab() {
    Tab t{};
    auto add = [&](int cls, int type, int a) { t.d[cls][t.n[cls]++] = (unsigned char)(type << 5 | a); };
    {
        const int early[4] = {8, 5, 4, 2};
        int rf = 0;
        for (int s = 0; s < 8; ++s) {
            if (s >= NPUSH) add(0, T_XR, s);
            for (int ph = 0; ph < 4; ++ph)
                for (int k = s * early[ph] / 8; k < (s + 1) * early[ph] / 8; ++k) add(0, T_W, k * 4 + ph);
            for (; s >= 1 && rf < NREL && rf < 2 * s; ++rf) add(0, T_RF, rf);
        }
        const int late[13][2] = {{5, 1}, {2, 3}, {6, 1}, {7, 1}, {4, 2}, {3, 3}, {5, 2}, {6, 2}, {7, 2}, {4, 3}, {5, 3}, {6, 3}, {7, 3}};
        for (auto& l : late) add(0, T_W, l[0] * 4 + l[1]);
        for (int s = 0; s < 8; ++s) add(0, T_XO, s);
    }
    for (int k = 0; 4 * k < NREL; ++k) add(1, T_RS, k);
    for (int s = 0; s < 8; ++s) {
        if (s >= NPUSH) add(1, T_XR, s);
        for (int ph = 3; ph >= 0; --ph) if (s * 4 + ph >= NREL) add(1, T_W, s * 4 + ph);
        if (s >= 2 && s < 5) add(1, T_XO, s - 2);
    }
    for (int s = 3; s < 8; ++s) add(1, T_XO, s);
    for (int a = 0; a < 16; ++a) add(1, T_OUT, a);
    return t;
}

}

extern "C" void kernel_launch(int rank, int nranks, ncclComm_t comm,
                              hipStream_t stream,
                              void* const* d_in, const int* in_sizes, int n_in,
                              void* d_out, int out_size,
                              const int* in_dtypes, int out_dtype,
                              void* const* d_out_all,
                              void* const* d_ws_all, size_t ws_bytes) {
    if (nranks != NR || (unsigned)rank >= (unsigned)NR || ws_bytes < WS_NEED || n_in < 2 ||
        in_sizes[0] != MM * KS || in_sizes[1] != KS * NN || out_size != (MM / NR) * NN) {
        fprintf(stderr, "kernel_launch: unexpected shapes\n");
        return;
    }
    HIP_CHECK(hipFuncSetAttribute((const void*)fused_kernel, hipFuncAttributeMaxDynamicSharedMemorySize, LDS_BYTES));
    Params p;
    p.x = (const float*)d_in[0];
    p.w = (const float*)d_in[1];
    p.out = (float*)d_out;
    for (int i = 0; i < NR; ++i) p.ws[i] = (char*)d_ws_all[i];
    p.rank = rank; p.pad = 0;
    p.lws = p.ws[rank];
    const Tab tab = make_tab();
    for (int i = 0; i < 2; ++i) { p.ntab[i] = tab.n[i]; for (int j = 0; j < 64; ++j) p.tab[i * 64 + j] = tab.d[i][j]; }
    fused_kernel<<<GRID, BLOCK, LDS_BYTES, stream>>>(p);
    HIP_CHECK(hipGetLastError());
}
```

```cpp
#include <hip/hip_runtime.h>
#include <rccl/rccl.h>
#include <cstdio>
#include <cstdint>
#include <cstddef>
#pragma clang diagnostic ignored "-Wint-to-pointer-cast"

#define HIP_CHECK(expr) do { hipError_t _e = (expr); if (_e != hipSuccess) { \
    fprintf(stderr, "HIP error %s at %s:%d\n", hipGetErrorString(_e), __FILE__, __LINE__); \
    return; } } while (0)

typedef unsigned u32x8 __attribute__((ext_vector_type(8)));
typedef unsigned u32x4 __attribute__((ext_vector_type(4)));
typedef unsigned u32x2 __attribute__((ext_vector_type(2)));
typedef unsigned short u16x2 __attribute__((ext_vector_type(2)));
typedef float f32x4 __attribute__((ext_vector_type(4)));
typedef float f32x2 __attribute__((ext_vector_type(2)));
typedef _Float16 f16x8 __attribute__((ext_vector_type(8)));
typedef _Float16 f16x2 __attribute__((ext_vector_type(2)));
typedef unsigned long long u64;

namespace {

constexpr int NR = 8;
constexpr int MM = 4096, NN = 4096;
constexpr int KS = 512;
constexpr int NSTEP = 64;

constexpr unsigned X_STEP = 16384, X_RB = X_STEP * NSTEP, X_BYTES = 8 * X_RB;
constexpr unsigned W_STEP = 8192, W_CG = W_STEP * NSTEP, W_BYTES = 32 * W_CG;
constexpr unsigned PK = 4352;
constexpr unsigned OFF_EX = X_BYTES + W_BYTES;
constexpr unsigned OFF_RX = OFF_EX + 2048 * PK;
constexpr unsigned PO = 4608;
constexpr unsigned OFF_OX = OFF_RX + 256 * PK;
constexpr unsigned OFF_W2 = OFF_OX + 1024 * PO;
constexpr unsigned OFF_F2 = OFF_W2 + 256 * W_STEP;
constexpr unsigned OFF_FX = OFF_F2 + 2048;
constexpr unsigned OFF_FW = OFF_FX + 8 * 64 * 16;
constexpr unsigned OFF_FE = OFF_FW + 32 * 64 * 8;
constexpr unsigned OFF_FR = OFF_FE + 8 * 1024;
constexpr unsigned OFF_FD = OFF_FR + 4096;
constexpr unsigned OFF_GO = OFF_FD + 4096;
constexpr unsigned OFF_TK = OFF_GO + 1024;
constexpr unsigned OFF_PL = OFF_TK + 1024;
constexpr unsigned PAR_BYTES = OFF_PL + 1024;
constexpr unsigned OFF_CT = 2 * PAR_BYTES + 1024;
constexpr unsigned OFF_DBG = OFF_CT + 2048 * 16;
constexpr size_t WS_NEED = OFF_DBG + 256 * 8 * 64 * 8;

constexpr int GRID = 256, BLOCK = 512;
#define TAU 1740
#define TAUP 1900
#define TAUPC 1900
#define CREDIT 0
#define LATE 40
#define LATE1 10
#define LATE2 20
#define EARLY 1
#define PRIO 1
#define STAG 0
#define RAWL 1
#define WARM 0
#define INT 1
#define S4 " sc0 sc1"
#define SP " sc0 sc1"
#define GOK 2
#define HOLDO 1200
#define HOLDR 0
#define PLAN_AT (FLEX_AT[0] * 24 - 220)
#define PTAIL 8
#define PNAP 24
#define PNAP2 2
#define ANAP 1
#define NAP 0
#define FMODE 0
constexpr int NPUSH = 2;
constexpr int NFLEX = 12, FLEX_AT[2] = {84, 82};
constexpr int NTAB[2] = {184, 206};
constexpr unsigned short TAB[2][256] = {
    {0, 1, 2, 16, 3, 17, 4, 18, 32, 5, 19, 33, 6, 20, 34, 48, 7, 21, 35, 49,
     8, 22, 36, 50, 9, 23, 544, 545, 546, 547, 560, 561, 562, 563, 37, 51, 10, 24, 38, 52,
     11, 25, 39, 576, 577, 578, 579, 592, 593, 594, 595, 608, 64, 609, 65, 610, 66, 80, 611, 624,
     53, 67, 81, 12, 625, 26, 40, 54, 68, 82, 96, 626, 13, 27, 41, 55, 69, 83, 97, 627,
     14, 28, 42, 56, 70, 84, 98, 112, 15, 29, 43, 57, 71, 85, 99, 113, 30, 44, 58, 72,
     86, 100, 114, 31, 45, 59, 73, 87, 101, 115, 46, 60, 74, 88, 102, 116, 47, 61, 75, 89,
     103, 117, 62, 76, 90, 104, 118, 63, 77, 91, 105, 119, 78, 92, 106, 120, 79, 93, 107, 121,
     94, 108, 122, 1024, 95, 1025, 109, 123, 1026, 1027, 1040, 110, 124, 1041, 1042, 1043, 111, 125, 1056, 1057,
     1058, 126, 1059, 1072, 1073, 127, 1074, 1075, 1088, 1089, 1090, 1091, 1104, 1105, 1106, 1107, 1120, 1121, 1122, 1123,
     1136, 1137, 1138, 1139},
    {1536, 1552, 1568, 1584, 1600, 1616, 1632, 1648, 1664, 1680, 1696, 1712, 48, 49, 50, 51, 52, 53, 54, 55,
     56, 57, 58, 59, 60, 61, 1024, 1025, 1026, 1027, 1040, 1041, 1042, 1043, 1056, 545, 546, 1057, 547, 1058,
     561, 1059, 562, 563, 1072, 577, 1073, 64, 65, 66, 67, 68, 578, 69, 1074, 579, 80, 70, 81, 1075,
     593, 71, 82, 1088, 594, 72, 83, 595, 62, 1089, 73, 84, 63, 609, 74, 1090, 85, 96, 610, 75,
     86, 1091, 97, 611, 76, 87, 98, 1104, 77, 625, 88, 99, 626, 78, 1105, 89, 100, 627, 79, 90,
     1106, 101, 112, 91, 102, 1107, 113, 92, 103, 114, 1120, 93, 104, 115, 1121, 94, 105, 116, 95, 1122,
     106, 117, 107, 1123, 118, 108, 119, 1136, 109, 120, 1137, 110, 121, 111, 1138, 122, 123, 1139, 124, 125,
     126, 127, 2560, 2561, 2562, 2563, 2576, 2577, 2578, 2579, 2592, 2593, 2594, 2595, 2608, 2609, 2610, 2611, 2624, 2625,
     2626, 2627, 2640, 2641, 2642, 2643, 2656, 2657, 2658, 2659, 2672, 2673, 2674, 2675, 2688, 2689, 2690, 2691, 2704, 2705,
     2706, 2707, 2720, 2721, 2722, 2723, 2736, 2737, 2738, 2739, 2752, 2753, 2754, 2755, 2768, 2769, 2770, 2771, 2784, 2785,
     2786, 2787, 2800, 2801, 2802, 2803},
};
constexpr int NWP = 64;
constexpr unsigned long long DEADLINE_TICKS = 50000000ull;

constexpr int LDS_RING = 6 * 24576;
constexpr int LDS_PACK = LDS_RING + 512;
constexpr int LDS_LUT = LDS_PACK + 8 * 640;
constexpr int LDS_BYTES = LDS_LUT + 2048;
typedef volatile __attribute__((address_space(3))) unsigned* ldsw_t;
typedef __attribute__((address_space(3))) void* ldsp_t;
typedef const __attribute__((address_space(1))) void* glbp_t;
enum { C_TZ = 0, C_CLK = 2, C_PCLK = 3, C_BAD = 4, C_PACKED = 5, C_PUSHED = 6, C_FULL = 8, C_PT = 15, C_DONE = 16, C_PACKW = 22, C_ORDER = 32 };

struct Params {
    const float* x;
    const float* w;
    char* lws;
    int rank, pad;
    float* out;
    char* ws[NR];
    int ntab[2];
    unsigned short tab[2 * 256];
};

__device__ __forceinline__ void store16_sys(void* p, u32x4 v) {
    asm volatile("global_store_dwordx4 %0, %1, off sc0 sc1\n\ts_nop 1" :: "v"(p), "v"(v) : "memory");
}
__device__ __forceinline__ void store4k_sys(char* base, unsigned off, u32x4 a, u32x4 b, u32x4 c, u32x4 d) {
    asm volatile("global_store_dwordx4 %0, %1, %5" S4 "\n\t"
                 "global_store_dwordx4 %0, %2, %5 offset:1024" S4 "\n\t"
                 "global_store_dwordx4 %0, %3, %5 offset:2048" S4 "\n\t"
                 "global_store_dwordx4 %0, %4, %5 offset:3072" S4 "\n\ts_nop 1"
                 :: "v"(off), "v"(a), "v"(b), "v"(c), "v"(d), "s"(base) : "memory");
}
__device__ __forceinline__ void store_packed(char* base, unsigned off, u32x4 a, u32x4 b, u32x4 c, u32x4 d) {
    u64 sv;
    asm volatile("global_store_dwordx4 %1, %2, %6" SP "\n\t"
                 "global_store_dwordx4 %1, %3, %6 offset:1024" SP "\n\t"
                 "global_store_dwordx4 %1, %4, %6 offset:2048" SP "\n\t"
                 "s_mov_b64 %0, exec\n\ts_mov_b32 exec_hi, 0\n\t"
                 "global_store_dwordx4 %1, %5, %6 offset:3072" SP "\n\ts_mov_b64 exec, %0\n\ts_nop 1"
                 : "=&s"(sv) : "v"(off), "v"(a), "v"(b), "v"(c), "v"(d), "s"(base) : "memory");
}
__device__ __forceinline__ void flag_sys(char* base, unsigned off, unsigned v) {
    u64 sv;
    asm volatile("s_mov_b64 %0, exec\n\ts_mov_b64 exec, 1\n\tglobal_store_dword %1, %2, %3 sc0 sc1\n\ts_mov_b64 exec, %0"
                 : "=&s"(sv) : "v"(off), "v"(v), "s"(base) : "memory");
}
__device__ __forceinline__ void wait_vm0() { asm volatile("s_waitcnt vmcnt(0)" ::: "memory"); }

template <class T> __device__ __forceinline__ T* uniform_ptr(T* p) {
    const u64 v = (u64)p;
    return (T*)(((u64)(unsigned)__builtin_amdgcn_readfirstlane((unsigned)(v >> 32)) << 32) | (u64)(unsigned)__builtin_amdgcn_readfirstlane((unsigned)v));
}

__device__ __forceinline__ unsigned now32() { return (unsigned)__builtin_amdgcn_s_memrealtime(); }
__device__ __forceinline__ void wait_until(unsigned t) {
    while ((int)(now32() - t) < 0) __builtin_amdgcn_s_sleep(1);
}

__device__ __forceinline__ unsigned pack2(float a, float b) {
    f32x2 v = {a, b};
    return __builtin_bit_cast(unsigned, __builtin_convertvector(v, f16x2));
}
__device__ __forceinline__ f32x2 gate(f32x2 y, float one) {
    constexpr float L2E = 1.44269504088896341f;
    f32x2 t = y * -L2E;
#pragma unroll
    for (int i = 0; i < 2; ++i) t[i] = __builtin_amdgcn_exp2f(__builtin_amdgcn_fmed3f(t[i], -60.f * L2E, 60.f * L2E));
    t += one;
    return f32x2{__builtin_amdgcn_rcpf(t[0]), __builtin_amdgcn_rcpf(t[1])};
}
__device__ __forceinline__ int wave_sum(unsigned v) {
    v += __builtin_amdgcn_update_dpp(0u, v, 0x111, 0xf, 0xf, true);
    v += __builtin_amdgcn_update_dpp(0u, v, 0x112, 0xf, 0xf, true);
    v += __builtin_amdgcn_update_dpp(0u, v, 0x114, 0xf, 0xf, true);
    v += __builtin_amdgcn_update_dpp(0u, v, 0x118, 0xf, 0xf, true);
    return __builtin_amdgcn_readlane(v, 15) + __builtin_amdgcn_readlane(v, 31) + __builtin_amdgcn_readlane(v, 47) + __builtin_amdgcn_readlane(v, 63);
}
__device__ __forceinline__ unsigned wave_scan(unsigned v) {
    v += __builtin_amdgcn_update_dpp(0u, v, 0x111, 0xf, 0xf, true);
    v += __builtin_amdgcn_update_dpp(0u, v, 0x112, 0xf, 0xf, true);
    v += __builtin_amdgcn_update_dpp(0u, v, 0x114, 0xf, 0xf, true);
    v += __builtin_amdgcn_update_dpp(0u, v, 0x118, 0xf, 0xf, true);
    v += __builtin_amdgcn_update_dpp(0u, v, 0x142, 0xa, 0xf, false);
    v += __builtin_amdgcn_update_dpp(0u, v, 0x143, 0xc, 0xf, false);
    return v;
}
__device__ __forceinline__ void store_planes(char* base, unsigned off, unsigned a, unsigned b, unsigned c) {
    asm volatile("global_store_dword %0, %1, %4 offset:3584 sc0 sc1\n\tglobal_store_dword %0, %2, %4 offset:3840 sc0 sc1\n\t"
                 "global_store_dword %5, %3, %4 offset:3840 sc0 sc1\n\ts_nop 1" :: "v"(off), "v"(a), "v"(b), "v"(c), "s"(base), "v"(off + 256) : "memory");
}
__device__ __forceinline__ unsigned bfi(unsigned m, unsigned a, unsigned b) { return __builtin_amdgcn_bitop3_b32(m, a, b, 0xca); }

#define TICKS(v) do {} while (0)
#define TICK(v) do {} while (0)
#define STAMPV(idx, v) do {} while (0)
#define STAMPN(idx, n) do {} while (0)
#define STAMP(idx) do {} while (0)

__device__ __forceinline__ unsigned lds_get(ldsw_t w) { return __builtin_amdgcn_readfirstlane(*w); }
__device__ __forceinline__ void lds_wait(ldsw_t w, unsigned v) {
    while (lds_get(w) < v) __builtin_amdgcn_s_sleep(1);
    asm volatile("" ::: "memory");
}
#define GAUX 17
__device__ __forceinline__ void glds4k(const char* g, char* l) {
    __builtin_amdgcn_global_load_lds((glbp_t)g, (ldsp_t)l, 16, 0, GAUX);
    __builtin_amdgcn_global_load_lds((glbp_t)g, (ldsp_t)l, 16, 1024, GAUX);
    __builtin_amdgcn_global_load_lds((glbp_t)g, (ldsp_t)l, 16, 2048, GAUX);
    __builtin_amdgcn_global_load_lds((glbp_t)g, (ldsp_t)l, 16, 3072, GAUX);
}
__device__ __forceinline__ unsigned lds_add(ldsw_t w, unsigned v) {
    unsigned r; u64 sv;
    asm volatile("s_mov_b64 %1, exec\n\ts_mov_b64 exec, 1\n\tds_add_rtn_u32 %0, %2, %3\n\ts_mov_b64 exec, %1\n\ts_waitcnt lgkmcnt(0)"
                 : "=&v"(r), "=&s"(sv) : "v"(w), "v"(v) : "memory");
    return __builtin_amdgcn_readfirstlane(r);
}
__device__ __forceinline__ void pace(ldsw_t clk, unsigned tz, unsigned d) {
    unsigned slot; u64 sv;
    asm volatile("s_mov_b64 %1, exec\n\ts_mov_b64 exec, 1\n\tds_max_u32 %2, %3\n\tds_add_rtn_u32 %0, %2, %4\n\t"
                 "s_mov_b64 exec, %1\n\ts_waitcnt lgkmcnt(0)"
                 : "=&v"(slot), "=&s"(sv) : "v"(clk), "v"((now32() - tz) << 5), "v"(d) : "memory");
    wait_until(tz - CREDIT + ((unsigned)__builtin_amdgcn_readfirstlane(slot) >> 5));
}
__device__ __forceinline__ int phase_cg(int ph, int xx, int c1) { return 8 * xx + 4 * (ph & 1) + (ph & 2) + c1; }

__device__ __forceinline__ int flex_row(int f) { return FMODE == 0 ? f : (f >> 1) * 4 + 2 + (f & 1); }

#define LIST_TAIL "v_mbcnt_lo_u32_b32 %4, exec_lo, %1\n\tv_mbcnt_hi_u32_b32 %4, exec_hi, %4\n\ts_bcnt1_i32_b64 %6, exec\n\t" \
    "v_min_u32_e32 %4, 63, %4\n\ts_add_u32 %2, %2, %6\n\tv_lshl_add_u32 %4, %4, 1, %11\n\ts_cmp_gt_u32 %2, 63\n\tds_write_b16 %4, %3\n\t" \
    "s_cbranch_scc1 2f\n\tv_mov_b32_e32 %1, %2\n\tv_cmpx_ne_u32_e32 vcc, 0, %0\n\ts_cbranch_execnz 1b\n2:\n\ts_mov_b64 exec, %7"

__device__ __forceinline__ unsigned pack_role(const Params& p, char* lds) {
    TICK(tk0);
    u32x8 arg;
    u64 tz, wsq;
    asm volatile("s_load_dwordx8 %0, %3, 0x0\n\ts_load_dwordx2 %1, %3, %4\n\ts_memrealtime %2" : "=&s"(arg), "=&s"(wsq), "=&s"(tz)
                 : "s"(__builtin_amdgcn_kernarg_segment_ptr()), "s"((unsigned)offsetof(Params, ws) + ((((blockIdx.x >> 2) & 4) ^ 4) + (blockIdx.x & 3)) * 8));
    const int lane = threadIdx.x & 63;
    const int wave = __builtin_amdgcn_readfirstlane(threadIdx.x >> 6);
    const ldsw_t ctl = (ldsw_t)(lds + LDS_RING);
    const int sub = wave & 3, kh = wave & 1;
    const int b0 = blockIdx.x < 128 ? blockIdx.x : 128 + (wave >> 2) * 32 + ((blockIdx.x & 63) >> 1);
    const int hf = blockIdx.x < 128 ? wave >> 2 : blockIdx.x & 1;
    f32x4 v[8];
    unsigned so, o0, o1, sa, sb;
    auto where = [&](int b) {
        if (b < 128) {
            so = (((b & 31) * 128 + sub * 32) * KS + (2 * (b >> 5) + hf) * 64) * 4;
            o0 = (lane >> 3) * KS * 4 + ((lane & 7) ^ (lane >> 4)) * 16; o1 = (o0 ^ 64) + 8 * KS * 4; sa = 128; sb = 16 * KS * 4;
        } else {
            so = ((((b >> 4) & 7) * 64 + kh * 16) * NN + ((b >> 3) & 1) * 2048 + phase_cg((b >> 1) & 3, 2 * (b & 1) + hf, (wave >> 1) & 1) * 64) * 4;
            o0 = ((lane >> 4) * NN + (lane & 15)) * 16; o1 = o0 + 2 * NN * 4; sa = NN * 4; sb = 32 * NN * 4;
        }
    };
    auto fetch = [&](const char* base) {
#pragma unroll
        for (int jj = 0; jj < 8; ++jj) {
            auto* a = (const __attribute__((address_space(1))) f32x4*)(base + ((jj & 2 ? o1 : o0) + (jj & 1) * sa + (jj >> 2) * sb));
            v[jj] = INT ? __builtin_nontemporal_load(a) : *a;
        }
        asm volatile("" ::: "memory");
    };
    {
        unsigned sel = 0, x = (threadIdx.x >> 1) & ((threadIdx.x >> 1) - (threadIdx.x & 1));
#pragma unroll
        for (int j = 0; j < 4; ++j) {
            sel |= __builtin_ctz(x | 0x1000) << 8 * j;
            x &= x - 1; x &= x - 1;
        }
        *(unsigned*)(lds + LDS_LUT + threadIdx.x * 4) = sel;
    }
    where(b0);
    asm volatile("" :: "s"(so), "v"(o0), "v"(o1), "s"(sa), "s"(sb));
    asm volatile("s_waitcnt lgkmcnt(0)" : "+s"(arg), "+s"(wsq), "+s"(tz));
    const char* const px = (const char*)((u64)arg[1] << 32 | arg[0]);
    const char* const pw = (const char*)((u64)arg[3] << 32 | arg[2]);
    char* const lws = (char*)((u64)arg[5] << 32 | arg[4]);
    const int rank = arg[6];
    TICK(tk1);
    if (wave >= 6) ctl[threadIdx.x & 127] = (threadIdx.x & 127) == C_TZ ? (unsigned)tz : 0;
    unsigned cv; u64 wv;
    asm volatile("global_load_dword %0, %2, %3 sc0 sc1\n\tglobal_load_dwordx2 %1, %4, %5" : "=&v"(cv), "=&v"(wv)
                 : "v"(OFF_CT + (blockIdx.x * 8 + wave) * 16), "s"(lws), "v"((unsigned)offsetof(Params, ws) + (lane & 7) * 8), "s"(__builtin_amdgcn_kernarg_segment_ptr()) : "memory");
    if (wave >= EARLY) asm volatile("s_waitcnt lgkmcnt(0)\n\ts_barrier" ::: "memory");
    if (LATE && (blockIdx.x >> 6) == 1 && (!EARLY || wave < EARLY)) __builtin_amdgcn_s_sleep(LATE);
    if (LATE1 && blockIdx.x >= 32 * ((NPUSH + 1) / 2) && blockIdx.x < 64 && (!EARLY || wave < EARLY)) __builtin_amdgcn_s_sleep(LATE1);
    if (LATE2 && (b0 >= 160 || (NPUSH == 3 && (b0 >> 5) == 1 && hf)) && blockIdx.x < 192) __builtin_amdgcn_s_sleep(LATE2);
    if (STAG && blockIdx.x < 32) for (int i = 1; i < wave; ++i) __builtin_amdgcn_s_sleep(STAG);
    if (PRIO && b0 >= 128 && b0 < 160) __builtin_amdgcn_s_setprio(2);
    if (blockIdx.x < 192) fetch((blockIdx.x < 128 ? px : pw) + so);
    if (wave < EARLY) asm volatile("s_barrier" ::: "memory");
    TICK(tk2);
    unsigned call, po;
    if (blockIdx.x >= 192) {
        asm volatile("s_waitcnt vmcnt(0)\n\tv_readfirstlane_b32 %0, %1" : "=s"(call) : "v"(cv), "v"(wv));
        ++call;
        flag_sys(lws, OFF_CT + (blockIdx.x * 8 + wave) * 16, call);
        if (PRIO) __builtin_amdgcn_s_setprio(0);
        STAMPV(0, tk0); STAMP(3);
        return call;
    }
#pragma unroll 1
    for (int b = b0;;) {
    const bool isx = b < 128;
    const int R = b & 31, sp = b >> 5, q = R >> 2;
    const int hh = b & 1, ph = (b >> 1) & 3, g = (b >> 3) & 1;
    const int cg = phase_cg(ph, 2 * hh + hf, (wave >> 1) & 1);
    const int s = isx ? 2 * sp + hf : (b >> 4) & 7, t = rank * 8 + s;
    unsigned eu, d1, f1, fe;
    int q1; bool on1, fq;
    if (isx) {
        eu = (R * 4 + sub) * 8 + s;
        q1 = ((q & 4) ^ 4) + (R & 3);
        on1 = q1 == rank || (s < NPUSH && q != rank);
        d1 = (4 + (q & 3)) * X_RB + t * X_STEP + sub * 4096; f1 = OFF_FX + ((4 + (q & 3)) * 64 + t) * 16 + sub * 4;
        fq = lane == q1 || lane == q;
        fe = lane == q1 ? 144 + (q & 3) * 4 + sp : 128 + (R & 3) * 4 + sp;
    } else {
        eu = 1024 + (g * 32 + cg) * 16 + s * 2 + kh;
        q1 = rank;
        on1 = true;
        const bool mine = g == rank >> 2;
        d1 = (mine ? X_BYTES + cg * W_CG + t * W_STEP : OFF_W2 + (cg * 8 + s) * W_STEP) + kh * 4096;
        f1 = (mine ? OFF_FW + (cg * 64 + t) * 8 : OFF_F2 + (cg * 8 + s) * 8) + kh * 4;
        fq = lane >> 2 == g;
        fe = (s * 4 + ph) * 4 + 2 * hh + hf;
    }
    const bool on2 = isx && q == rank;
    const unsigned d2 = (R & 3) * X_RB + t * X_STEP + sub * 4096, f2 = OFF_FX + ((R & 3) * 64 + t) * 16 + sub * 4;
    char* fws;
    char* const ws1 = isx ? (char*)wsq : lws;
    char* const sc = lds + LDS_PACK + wave * 640;
    *(unsigned*)(sc + 320 + lane * 4) = 0;
    asm volatile("s_waitcnt vmcnt(8)\n\tv_readfirstlane_b32 %0, %2\n\tv_mov_b64 %1, %3" : "=s"(call), "=v"(fws) : "v"(cv), "v"(wv), "s"(eu), "s"(d1), "s"(f1), "v"(fe), "s"(d2), "s"(f2));
    ++call;
    po = (call & 1) * PAR_BYTES;

    unsigned h[16];
    if (isx) {
#pragma unroll
        for (int i = 0; i < 8; ++i) { h[2 * i] = pack2(v[i][0], v[i][1]); h[2 * i + 1] = pack2(v[i][2], v[i][3]); }
    } else {
#pragma unroll
        for (int i = 0; i < 16; ++i) h[i] = pack2(v[2 * (i & 3)][i >> 2], v[2 * (i & 3) + 1][i >> 2]);
    }
    asm volatile("" :: "v"(h[0]), "v"(h[15]));
    TICK(tk3);
    const u32x4 raw[4] = {{h[0], h[1], h[2], h[3]}, {h[4], h[5], h[6], h[7]}, {h[8], h[9], h[10], h[11]}, {h[12], h[13], h[14], h[15]}};
    const bool keep = on2 || q1 == rank;
    if (on1 && (!RAWL || q1 != rank)) {
        if (q1 != rank) {
            const unsigned k = lds_add(ctl + C_PUSHED, 1);
            pace(ctl + C_PCLK, lds_get(ctl + C_TZ), ((q ^ rank) & 4 ? TAUP * NPUSH / 4 : TAUPC * NPUSH / 4) << (2 * (b >> 5) + 2 > NPUSH));
            if (k == GOK && b < 32) flag_sys(ws1, po + OFF_GO + rank * 128 + (q & 3) * 4, call);
        }
        store4k_sys(ws1, po + d1 + lane * 16, raw[0], raw[1], raw[2], raw[3]);
    }
    if (!RAWL && on2) store4k_sys(lws, po + d2 + lane * 16, raw[0], raw[1], raw[2], raw[3]);
    TICK(tk4);
    if (!on2) {
    unsigned es = ((h[0] >> 7) & 0x00ff00ffu) + ((h[5] >> 7) & 0x00ff00ffu) + ((h[10] >> 7) & 0x00ff00ffu) + ((h[15] >> 7) & 0x00ff00ffu);
    es = (es & 0xffffu) + (es >> 16);
    const unsigned eb = min(max((wave_sum(es) >> 2) - 5530, 0), 0x6000);
    unsigned tn[4];
#pragma unroll
    for (int j = 0; j < 16; ++j) h[j] = __builtin_bit_cast(unsigned, __builtin_bit_cast(u16x2, h[j]) - __builtin_bit_cast(u16x2, eb * 0x00010001u));
#pragma unroll
    for (int k = 0; k < 4; ++k)
        tn[k] = bfi(0x000f000fu, h[4 * k] >> 12, bfi(0x00ff00ffu, h[4 * k + 1] >> 8, bfi(0x0fff0fffu, h[4 * k + 2] >> 4, h[4 * k + 3])));
    const unsigned a0 = bfi(0x55555555u, tn[0], tn[1] << 1), a2 = bfi(0x55555555u, tn[2], tn[3] << 1);
    const unsigned p12 = ~bfi(0x33333333u, a0, a2 << 2), p14 = bfi(0x33333333u, a0 >> 2, a2);
    const unsigned p13 = bfi(0x33333333u, bfi(0x55555555u, tn[0] >> 1, tn[1]), bfi(0x55555555u, tn[2] << 1, tn[3] << 2));
    const unsigned na = p12 & 0x0f0f0f0fu, nb = (p12 >> 4) & 0x0f0f0f0fu;
    const unsigned mk = __builtin_amdgcn_udot4((na + 0x0f0f0f0fu) & 0x10101010u, 0x08040201u,
                                               __builtin_amdgcn_udot4((nb + 0x0f0f0f0fu) & 0x10101010u, 0x80402010u, 0, false), false);
    const u32x2 sel = *(const u32x2*)(lds + LDS_LUT + (mk >> 1));
    *(unsigned char*)(sc + 256 + lane) = (unsigned char)(mk >> 4);
    *(unsigned*)(sc + lane * 4) = bfi(0x40004000u, h[13] >> 1, bfi(0x20002000u, h[14] >> 2, bfi(0x10001000u, h[15] >> 3, h[12])));
    const unsigned nn = __builtin_popcount(mk), upto = wave_scan(nn);
    int room = 182 - ((__builtin_amdgcn_readlane(upto, 63) + 1) >> 1);
    __builtin_amdgcn_sched_barrier(0);
    {
        const u64 c = (u64)(__builtin_amdgcn_perm(nb, na, sel[0]) | __builtin_amdgcn_perm(nb, na, sel[1]) << 4) << ((upto - nn) * 4 & 28);
        auto* d = (__attribute__((address_space(3))) unsigned*)(sc + 320 + ((upto - nn) >> 3) * 4);
        __hip_atomic_fetch_or(d, (unsigned)c, __ATOMIC_RELAXED, __HIP_MEMORY_SCOPE_WORKGROUP);
        __hip_atomic_fetch_or(d + 1, (unsigned)(c >> 32), __ATOMIC_RELAXED, __HIP_MEMORY_SCOPE_WORKGROUP);
    }
    u64 first;
    {
        unsigned x = p13 | p14, top = (unsigned)(size_t)(ldsp_t)(sc + 256 + 245), t0, t1, t2, n; u64 sv;
        asm volatile("s_mov_b64 %7, exec\n\tv_cmpx_ne_u32_e32 vcc, 0, %0\n\ts_mov_b64 %8, exec\n\ts_cbranch_execz 2f\n1:\n\t"
                     "v_ffbl_b32_e32 %3, %0\n\tv_add_u32_e32 %4, -1, %0\n\ts_bcnt1_i32_b64 %6, exec\n\tv_and_b32_e32 %0, %0, %4\n\t"
                     "v_bfe_u32 %4, %10, %3, 1\n\tv_bfe_u32 %5, %9, %3, 1\n\tv_cmp_ne_u32_e32 vcc, 0, %0\n\ts_sub_i32 %1, %1, %6\n\t"
                     "v_lshl_or_b32 %3, %3, 1, %4\n\tv_mbcnt_lo_u32_b32 %4, exec_lo, 0\n\tv_lshl_or_b32 %3, %3, 1, %5\n\t"
                     "v_mbcnt_hi_u32_b32 %4, exec_hi, %4\n\tv_addc_co_u32_e32 %3, vcc, %3, %3, vcc\n\ts_cmp_lt_i32 %1, 0\n\t"
                     "v_sub_u32_e32 %4, %2, %4\n\ts_cbranch_scc1 2f\n\tds_write_b8 %4, %3\n\ts_sub_u32 %2, %2, %6\n\t"
                     "v_cmpx_ne_u32_e32 vcc, 0, %0\n\ts_cbranch_execnz 1b\n2:\n\ts_mov_b64 exec, %7"
                     : "+v"(x), "+s"(room), "+s"(top), "=&v"(t0), "=&v"(t1), "=&v"(t2), "=&s"(n), "=&s"(sv), "=&s"(first)
                     : "v"(p13), "v"(p14) : "memory", "vcc", "scc");
    }
    asm volatile("" ::: "memory");
    if (lane == 0) {
        *(unsigned short*)(sc + 256 + 246) = (unsigned short)(eb | (room < 0 ? 0x8000 : 0));
        *(u64*)(sc + 256 + 248) = first;
    }
    asm volatile("" ::: "memory");
    const u32x4 tail = *(const u32x4*)(sc + (lane & 31) * 16);
    __builtin_amdgcn_sched_barrier(0);
#pragma unroll
    for (int j = 0; j < 12; ++j) h[j] = bfi(0x70007000u, h[13 + (j >> 2)] << (12 - 3 * (j & 3)), h[j]);
    store_packed(lws, po + OFF_EX + eu * PK + lane * 16, u32x4{h[0], h[1], h[2], h[3]}, u32x4{h[4], h[5], h[6], h[7]},
                 u32x4{h[8], h[9], h[10], h[11]}, tail);
    if (room < 0) store_planes(lws, po + OFF_EX + eu * PK + lane * 4, p12, p13, p14);
    }
    TICK(tk5);
    if (RAWL && keep) {
        store4k_sys(lws, po + (on2 ? d2 : d1) + lane * 16, raw[0], raw[1], raw[2], raw[3]);
        asm volatile("s_waitcnt vmcnt(4)" ::: "memory");
    } else {
        wait_vm0();
    }
    if (lds_add(ctl + (isx ? C_PACKED : C_PACKW + (wave >> 2)), 1) == (isx ? 7 : 3) && fq)
        asm volatile("global_store_dword %0, %1, off sc0 sc1" :: "v"(fws + po + OFF_FE + rank * 1024 + fe * 4), "v"(call) : "memory");
    TICK(tk6);
    if (RAWL && keep) wait_vm0();
    if (on1) flag_sys(ws1, po + f1, call);
    if (on2) flag_sys(lws, po + f2, call);
    STAMPV(0, tk0); STAMPV(7, tk5); STAMPV(5, tk2); STAMPV(3, tk3); STAMPV(4, tk4); STAMPV(6, tk6);
    if ((b >> 6) != 1) break;
    b |= 128;
    where(b);
    fetch(pw + so);
    }
    flag_sys(lws, OFF_CT + (blockIdx.x * 8 + wave) * 16, call);
    if (PRIO) __builtin_amdgcn_s_setprio(0);
    return call;
}

enum { T_W = 0, T_XR = 1, T_XO = 2, T_RS = 3, T_RF = 4, T_OUT = 5 };
enum { K_IN = 0, K_RS = 1, K_OUT = 2 };
struct Task { int kind; unsigned src, wf, dst, fl, rel; };

__device__ __forceinline__ void pull_role(const Params& p, char* lds, int e, int xcd, unsigned call) {
    const int lane = threadIdx.x & 63;
    const int wave = __builtin_amdgcn_readfirstlane(threadIdx.x >> 6);
    const int jw = wave * 8 + xcd;
    char* const lb = lds + wave * 16384;
    const ldsw_t ctl = (ldsw_t)(lds + LDS_RING);
    const int rank = p.rank, q = rank ^ e;
    const int cls = e >> 2;
    u32x2 desc = *(const u32x2*)(p.tab + cls * 256 + lane * 4);
    const int ntask = p.ntab[cls] * 4;
    const unsigned po = (call & 1) * PAR_BYTES;
    const char* const sws = p.ws[q] + po;
    char* const lws = p.lws;
    char* const pws = p.ws[rank ^ (lane % 3 + 1)] + po + OFF_FR + (rank & 3) * 1024;
    const unsigned tz = lds_get(ctl + C_TZ);

    {
        const unsigned mk = threadIdx.x >> 1, w = threadIdx.x & 1;
        unsigned sel = 0;
#pragma unroll
        for (int j = 0; j < 4; ++j) {
            const unsigned k = __builtin_popcount(mk & ((1u << (4 * w + j)) - 1));
            sel |= ((mk >> (4 * w + j)) & 1 ? (k >> 1) + (k & 1) * 4 : 12) << 8 * j;
        }
        *(unsigned*)(lds + LDS_LUT + threadIdx.x * 4) = sel;
        asm volatile("s_waitcnt lgkmcnt(0)\n\ts_barrier" ::: "memory");
    }
    auto decode = [&](int u, int fx, Task& tk) {
        const unsigned d0 = __builtin_amdgcn_readlane(desc[0], u >> 4), d1 = __builtin_amdgcn_readlane(desc[1], u >> 4);
        int d = ((u & 8 ? d1 : d0) >> (u & 4) * 4) & 0xffff, j = u & 3;
        if (fx >= 0) { j = (fx >> 2) & 3; d = cls ? flex_row(fx >> 4) * 4 + (fx & 3) : T_RF << 9 | (fx & ~12); }
        const int type = d >> 9, a = (d >> 4) & 31, k = d & 15;
        tk.kind = K_IN; tk.rel = 0;
        if (type == T_XR || type == T_XO) {
            const int rbl = (k + q) & 3, t = q * 8 + a;
            const int row = type == T_XO ? rank * 16 + rbl * 4 + j : (((rank & 4) ^ 4) + rbl) * 16 + (rank & 3) * 4 + j;
            const int rb = type == T_XO ? rbl : 4 + rbl;
            tk.src = OFF_EX + (row * 8 + a) * PK;
            tk.wf = OFF_FE + q * 1024 + (128 + rb * 4 + (a >> 1)) * 4;
            tk.dst = rb * X_RB + t * X_STEP + j * 4096;
            tk.fl = OFF_FX + (rb * 64 + t) * 16 + j * 4;
        } else if (type == T_OUT) {
            const int pair = phase_cg(a >> 2, a & 3, 0) >> 1;
            tk.kind = K_OUT;
            tk.src = OFF_OX + (((rank & 3) * 16 + pair) * 16 + j * 4 + k) * PO;
            tk.wf = OFF_FD + (((q & 3) * 16 + pair) * 16 + k * 4 + j) * 4;
            tk.dst = (((q & 3) * 128 + (j >> 1) * 64 + k * 16) * NN + (q >> 2) * 2048 + pair * 128 + (j & 1) * 64) * 4;
            tk.fl = 160 + 2 * pair + (j & 1);
        } else {
            const int r = type == T_W ? 4 * a + (k >> 2) : flex_row(a), s = r >> 2, ph = r & 3, kh = j & 1;
            const int xx = (type == T_W ? k : (type == T_RS ? rank : q)) & 3;
            const int cg = phase_cg(ph, xx, j >> 1);
            const int t = (type == T_RF ? ((rank & 4) ^ 4) + k : q) * 8 + s;
            tk.rel = ((type == T_RF ? k : q & 3) * 16 + (a & 15)) * 4 + j;
            tk.src = type == T_RF ? OFF_RX + tk.rel * PK : OFF_EX + (1024 + ((rank >> 2) * 32 + cg) * 16 + s * 2 + kh) * PK;
            tk.wf = type == T_RF ? OFF_FR + (q & 3) * 1024 + tk.rel * 4 : OFF_FE + q * 1024 + (r * 4 + xx) * 4;
            tk.dst = X_BYTES + cg * W_CG + t * W_STEP + kh * 4096;
            tk.fl = OFF_FW + (cg * 64 + t) * 8 + kh * 4;
            if (type == T_RS) tk.kind = K_RS;
        }
    };
    auto await = [&](unsigned off) {
        const unsigned* fl = (const unsigned*)(lws + po + off);
        for (;;) {
            if (!__builtin_amdgcn_ballot_w64(__hip_atomic_load(fl, __ATOMIC_RELAXED, __HIP_MEMORY_SCOPE_SYSTEM) < call)) return true;
            if ((int)((unsigned)__builtin_amdgcn_s_memrealtime() - tz) > (int)DEADLINE_TICKS) return false;
            __builtin_amdgcn_s_sleep(ANAP);
        }
    };

    asm volatile("" : "+v"(desc) :: "memory");
    Task cur;
    int un = jw;
    int fx = -1;
    u32x2 plan = {0, 0};
    const unsigned fa = OFF_FE + q * 1024, fb = (cls ? OFF_FD : OFF_FR) + (q & 3) * 1024;
    u32x4 sa = {0, 0, 0, 0}, sb = {0, 0, 0, 0};
#pragma unroll 1
    for (int i = 0;; ++i) {
        decode(un, fx, cur);
        const bool isa = cur.wf < OFF_FR;
        const unsigned wi = (cur.wf - (isa ? fa : fb)) >> 2;
        const bool out = cur.kind == K_OUT;
        const unsigned oo = cur.dst + (out ? ((lane >> 4) * 4 * NN + (lane & 15) * 4) * 4 : po + lane * 16);
        auto pick = [](u32x4 f, unsigned i) { return (unsigned)__builtin_amdgcn_readlane(i & 2 ? (i & 1 ? f[3] : f[2]) : (i & 1 ? f[1] : f[0]), i >> 2); };
        unsigned seen = isa || out ? pick(sa, out ? cur.fl : wi) : call;
        if (!isa) seen = min(seen, pick(sb, wi));
        bool ok = true;
        if (seen < call) ok = await(NPUSH && i == 0 && lane < 4 && lane + 4 != (q ^ (rank & 4)) ? OFF_GO + q * 128 + lane * 4 : out && lane == 4 ? fa + cur.fl * 4 : cur.wf);
        asm volatile("" ::: "memory");
        if (ok) {
            pace(ctl + C_CLK, tz, TAU * 28 / 32);
            const char* g = sws + cur.src + lane * 16;
            __builtin_amdgcn_global_load_lds((glbp_t)g, (ldsp_t)lb, 16, 0, 17);
            __builtin_amdgcn_global_load_lds((glbp_t)g, (ldsp_t)lb, 16, 1024, 17);
            __builtin_amdgcn_global_load_lds((glbp_t)g, (ldsp_t)lb, 16, 2048, 17);
            if (lane < 32) __builtin_amdgcn_global_load_lds((glbp_t)g, (ldsp_t)lb, 16, 3072, 17);
            if (out) {
#pragma unroll
                for (int r = 0; r < 4; ++r) __builtin_amdgcn_global_load_lds((glbp_t)((const char*)p.out + (oo + r * NN * 4)), (ldsp_t)(lb + 8192 + r * 1024), 16, 0, 17);
            }
        }
        TICK(ts0);
        wait_vm0();
        TICK(ts1);
        unsigned h[16];
        u32x4 m[3];
        const unsigned mk = *(const unsigned char*)(lb + 3328 + lane);
        const u32x4 hd = *(const u32x4*)(lb + 3328 + 240);
        __builtin_amdgcn_sched_barrier(0);
#pragma unroll
        for (int k = 0; k < 3; ++k) m[k] = *(const u32x4*)(lb + k * 1024 + lane * 16);
#pragma unroll
        for (int j = 0; j < 14; ++j) h[j] = j < 12 ? m[j >> 2][j & 3] : *(const unsigned*)(lb + 3072 + lane * (out ? 8 : 4) + (j - 12) * 4);
        __builtin_amdgcn_sched_barrier(0);
        if (out) {
            *(u32x4*)(lb + 3584 + lane * 16) = u32x4{0, 0, 0, 0};
            f32x4 z[4];
#pragma unroll
            for (int r = 0; r < 4; ++r) z[r] = *(const f32x4*)(lb + 8192 + r * 1024 + lane * 16);
            unsigned eh[2];
#pragma unroll
            for (int k = 0; k < 2; ++k) {
                unsigned d = h[7 * k + 6] << 4;
#pragma unroll
                for (int j = 5; j >= 0; --j) d = bfi((32u << 5 * j) - 1, h[7 * k + j] >> (26 - 5 * j), d);
                h[14 + k] = d;
                eh[k] = bfi(31, d >> 26, h[7 * k + 6] >> 23);
            }
            const unsigned hdr = __builtin_amdgcn_readlane(eh[0], 63) & 255;
            asm volatile("" ::: "memory");
            if ((__builtin_amdgcn_readlane(eh[1], 63) & 128) && ok) {
                pace(ctl + C_CLK, tz, TAU / 4);
                __builtin_amdgcn_global_load_lds((glbp_t)(sws + cur.src + lane * 16), (ldsp_t)lb, 16, 3584, 17);
                wait_vm0();
            } else if (eh[1] & 0x7c) {
                *(unsigned char*)(lb + 3584 + bfi(255, eh[0], (eh[1] & 3) << 8)) = (unsigned char)eh[1];
            }
            asm volatile("" ::: "memory");
            const u32x4 pl = *(const u32x4*)(lb + 3584 + lane * 16);
            float y[16];
#pragma unroll
            for (int c = 0; c < 16; ++c) y[c] = __builtin_bit_cast(float, bfi(0x7c000000u, pl[c >> 2] << (24 - 8 * (c & 3)), h[c]) + (hdr << 23));
#pragma unroll
            for (int c = 0; c < 16; c += 2) {
                f32x2 u = f32x2{y[(c & 3) * 4 + (c >> 2)], y[(c & 3) * 4 + 4 + (c >> 2)]} + f32x2{z[c >> 2][c & 3], z[c >> 2][(c & 3) + 1]};
                const u32x2 w = __builtin_bit_cast(u32x2, u * gate(u, ok ? 1.f : __builtin_nanf("")));
                h[c] = w[0];
                h[c + 1] = w[1];
            }
        } else if (ok) {
            const u32x2 sel = *(const u32x2*)(lds + LDS_LUT + mk * 8);
            const unsigned hdr = __builtin_amdgcn_readfirstlane(hd[1]);
            const unsigned f0 = __builtin_amdgcn_readfirstlane(hd[2]), f1 = __builtin_amdgcn_readfirstlane(hd[3]);
            unsigned ent = *(const unsigned char*)(lb + 3328 + 245 - __builtin_amdgcn_mbcnt_hi(f1, __builtin_amdgcn_mbcnt_lo(f0, 0)));
            __builtin_amdgcn_sched_barrier(0);
            const unsigned nn = __builtin_popcount(mk), at = wave_scan(nn) - nn;
            const unsigned* d = (const unsigned*)(lb + 3328 + 64 + (at >> 3) * 4);
            const unsigned d0 = d[0], d1 = d[1];
            __builtin_amdgcn_sched_barrier(0);
#pragma unroll
            for (int k = 0; k < 3; ++k)
                h[13 + k] = bfi(0x80008000u, h[12] << (1 + k), bfi(0x00070007u, h[4 * k] >> 12, bfi(0x003f003fu, h[4 * k + 1] >> 9,
                                bfi(0x01ff01ffu, h[4 * k + 2] >> 6, h[4 * k + 3] >> 3))));
            asm volatile("" : "+v"(h[13]), "+v"(h[14]), "+v"(h[15]));
            unsigned p13 = 0, p14 = 0;
            {
                unsigned top = (unsigned)(size_t)(ldsp_t)(lb + 3328 + 245), t1, t2, n; u64 sv;
                asm volatile("s_mov_b64 %7, exec\n\ts_mov_b64 exec, %8\n1:\n\t"
                             "s_bcnt1_i32_b64 %6, exec\n\tv_lshrrev_b32_e32 %4, 3, %3\n\tv_bfe_u32 %5, %3, 1, 1\n\tv_lshl_or_b32 %0, %5, %4, %0\n\t"
                             "v_bfe_u32 %5, %3, 2, 1\n\tv_and_b32_e32 %3, 1, %3\n\tv_lshl_or_b32 %1, %5, %4, %1\n\t"
                             "v_cmpx_ne_u32_e32 vcc, 0, %3\n\ts_sub_u32 %2, %2, %6\n\ts_cbranch_execz 2f\n\t"
                             "v_mbcnt_lo_u32_b32 %3, exec_lo, 0\n\tv_mbcnt_hi_u32_b32 %3, exec_hi, %3\n\t"
                             "v_sub_u32_e32 %3, %2, %3\n\tds_read_u8 %3, %3\n\ts_waitcnt lgkmcnt(0)\n\ts_branch 1b\n2:\n\ts_mov_b64 exec, %7"
                             : "+v"(p13), "+v"(p14), "+s"(top), "+v"(ent), "=&v"(t1), "=&v"(t2), "=&s"(n), "=&s"(sv)
                             : "s"((int)hdr < 0 ? 0 : (u64)f1 << 32 | f0) : "memory", "vcc", "scc");
            }
            const unsigned c = __builtin_amdgcn_alignbit(d1, d0, at * 4), na = c & 0x0f0f0f0fu, nb = (c >> 4) & 0x0f0f0f0fu;
            unsigned p12 = __builtin_amdgcn_perm(nb, na, sel[0]) | __builtin_amdgcn_perm(nb, na, sel[1]) << 4;
            if ((int)hdr < 0) {
                pace(ctl + C_CLK, tz, TAU * 6 / 32);
                if (lane < 48) __builtin_amdgcn_global_load_lds((glbp_t)(sws + cur.src + lane * 16), (ldsp_t)lb, 16, 3584, 17);
                wait_vm0();
                p12 = *(const unsigned*)(lb + 3584 + lane * 4); p13 = *(const unsigned*)(lb + 3840 + lane * 4); p14 = *(const unsigned*)(lb + 4096 + lane * 4);
            }
            if (cur.kind == K_RS) {
                store_packed(lws, po + OFF_RX + cur.rel * PK + lane * 16, m[0], m[1], m[2], *(const u32x4*)(lb + 3072 + lane * 16));
                if ((int)hdr < 0) store_planes(lws, po + OFF_RX + cur.rel * PK + lane * 4, p12, p13, p14);
            }
            const unsigned a0 = bfi(0x33333333u, ~p12, p14 << 2), a2 = bfi(0x33333333u, ~p12 >> 2, p14);
            const unsigned tn[4] = {bfi(0x55555555u, a0, p13 << 1), bfi(0x55555555u, a0 >> 1, p13), bfi(0x55555555u, a2, p13 >> 1), bfi(0x55555555u, a2 >> 1, p13 >> 2)};
#pragma unroll
            for (int j = 0; j < 16; ++j) {
                const unsigned t = bfi(0x70007000u, tn[j >> 2] << (12 - 4 * (j & 3)), h[j]);
                h[j] = __builtin_bit_cast(unsigned, __builtin_bit_cast(u16x2, t) + __builtin_bit_cast(u16x2, ((hdr >> 16) & 0x7fff) * 0x00010001u));
            }
        }
        unsigned cv; u64 sv;
        asm volatile("global_load_dwordx4 %1, %7, %9 sc0 sc1\n\tglobal_load_dwordx4 %2, %8, %9 sc0 sc1\n\t"
                     "s_mov_b64 %3, exec\n\ts_mov_b64 exec, 1\n\tglobal_atomic_add %0, %5, %6, %9 sc0\n\t"
                     "v_cmpx_ne_u32_e32 vcc, %10, %11\n\tglobal_load_dwordx2 %4, %5, %9 offset:1024 sc0 sc1\n\ts_mov_b64 exec, %3"
                     : "=&v"(cv), "=&v"(sa), "=&v"(sb), "=&s"(sv), "+&v"(plan)
                     : "v"(po + OFF_TK + e * 128), "v"(1u), "v"(po + fa + min(lane, 47) * 16), "v"(po + fb + lane * 16), "s"(lws), "s"(call), "v"(plan[0]) : "memory", "vcc");
        if (ok || out) {
            const unsigned o = oo, st = out ? NN * 4 : 1024;
            asm volatile("global_store_dwordx4 %0, %4, %8 sc0 sc1\n\tglobal_store_dwordx4 %1, %5, %8 sc0 sc1\n\t"
                         "global_store_dwordx4 %2, %6, %8 sc0 sc1\n\tglobal_store_dwordx4 %3, %7, %8 sc0 sc1\n\ts_nop 1"
                         :: "v"(o), "v"(o + st), "v"(o + 2 * st), "v"(o + 3 * st), "v"(u32x4{h[0], h[1], h[2], h[3]}), "v"(u32x4{h[4], h[5], h[6], h[7]}),
                            "v"(u32x4{h[8], h[9], h[10], h[11]}), "v"(u32x4{h[12], h[13], h[14], h[15]}), "s"(out ? (char*)p.out : lws) : "memory");
        }
        TICK(ts2);
        if (!out) wait_vm0();
        TICK(ts3);
        if (ok && !out) {
            flag_sys(lws, po + cur.fl, call);
            if (cur.kind == K_RS && lane < 3)
                asm volatile("global_store_dword %0, %1, off sc0 sc1" :: "v"(pws + cur.rel * 4), "v"(call) : "memory");
        }
        asm volatile("s_waitcnt vmcnt(4)" : "+v"(cv), "+v"(sa), "+v"(sb), "+v"(plan));
        un = __builtin_amdgcn_readfirstlane(cv) + NWP;
        fx = -1;
        const int v = un - (cls ? FLEX_AT[1] : FLEX_AT[0]) * 4;
        if (v >= 0) {
            if (__builtin_amdgcn_readfirstlane(plan[0]) != call) {
                if (!await(OFF_PL + e * 128)) break;
                asm volatile("global_load_dwordx2 %0, %1, %2 sc0 sc1\n\ts_waitcnt vmcnt(0)" : "=v"(plan) : "v"(po + OFF_PL + e * 128), "s"(lws) : "memory");
            }
            const unsigned cum = __builtin_amdgcn_readfirstlane(plan[1]);
            const int c = (v >= (int)(cum & 255)) + (v >= (int)((cum >> 8) & 255)) + (v >= (int)((cum >> 16) & 255));
            const int k = v - (int)((cum << 8 >> 8 * c) & 255);
            if (v < (int)(cum >> 24)) { fx = (cls ? 4 * NFLEX - 1 - k : k) * 4 + c; un = 0; }
            else un -= cum >> 24;
        }
        if (un >= ntask) break;
    }
    STAMP(1);
}

__device__ __forceinline__ void plan_role(const Params& p, char* lds, unsigned call) {
    const int e = threadIdx.x & 7;
    const unsigned po = (call & 1) * PAR_BYTES;
    asm volatile("global_store_dword %0, %1, %2 sc0 sc1" :: "v"(PAR_BYTES - po + OFF_TK + e * 128), "v"(0u), "s"(p.lws) : "memory");
    wait_until(lds_get((ldsw_t)(lds + LDS_RING) + C_TZ) + PLAN_AT);
    int a;
    asm volatile("s_mov_b64 exec, 255\n\tglobal_atomic_add %0, %1, %2, %3 sc0\n\ts_waitcnt vmcnt(0)"
                 : "=&v"(a) : "v"(po + OFF_TK + e * 128), "v"(0u), "s"(p.lws) : "memory");
    a -= (e & 4 ? FLEX_AT[1] : FLEX_AT[0]) * 4;
    int as[8], top = -4096, same = 0, all = 0;
#pragma unroll
    for (int i = 1; i < 8; ++i) { as[i] = __builtin_amdgcn_readlane(a, i); top = max(top, as[i]); }
#pragma unroll
    for (int i = 1; i < 8; ++i) { as[i] = max(as[i], top - 8 * NFLEX); all += as[i]; if (i < 4) same += as[i]; }
    as[0] = 0;
    a = max(a, top - 8 * NFLEX);
    unsigned cum = 0, n = 0;
#pragma unroll
    for (int c = 0; c < 4; ++c) {
        const int o = c ^ (p.rank & 3);
        const int av = e & 4 ? (o & 2 ? (o & 1 ? as[3] : as[2]) : as[1]) : a, at = e & 4 ? a : (o & 2 ? (o & 1 ? as[7] : as[6]) : (o & 1 ? as[5] : as[4]));
        const int x = min(max((int)((144 * NFLEX + 4 * all + 21 * av - 28 * at - 7 * same) * (1.f / 84) + 0.5f), 0), 4 * NFLEX);
        n += e & 4 ? (o ? 4 * NFLEX - x : 0) : x;
        cum |= n << 8 * c;
    }
    asm volatile("global_store_dwordx2 %0, %1, %2 sc0 sc1" :: "v"(po + OFF_PL + e * 128), "v"(u32x2{call, cum}), "s"(p.lws) : "memory");
}

template <int NB>
__device__ __forceinline__ void tile_role(const Params& p, char* lds, int rb, int cg, unsigned call) {
    constexpr int SLOT = 16384 + NB * 8192;
    constexpr int NS = LDS_RING / SLOT;
    constexpr int MT = 2 * NB;
    constexpr int NN_STEP = NB == 1 ? 8 + NSTEP : NSTEP - 8;
    const int lane = threadIdx.x & 63;
    const int wave = __builtin_amdgcn_readfirstlane(threadIdx.x >> 6);
    const int rank = p.rank;
    const unsigned po = (call & 1) * PAR_BYTES;
    char* ws = p.lws + po;
    const ldsw_t ctl = (ldsw_t)(lds + LDS_RING);

    if (wave == 4) {
        const unsigned* fx = (const unsigned*)(ws + OFF_FX + (rb * 64 + lane) * 16);
        const u64 t0 = __builtin_amdgcn_s_memrealtime();
        int cnt = 0, nb_dbg = 0;
        wait_until(lds_get(ctl + C_TZ) + (NB == 1 ? HOLDO : HOLDR));
#pragma unroll 1
        for (int pass = NB - 1; pass < 2; ++pass) {
        const unsigned* fw = (const unsigned*)(ws + (pass ? OFF_FW + cg * 512 : OFF_F2 + (cg - rank) * 64) + lane * 8);
        u32x4 a = {0, 0, 0, 0}; u32x2 b = {0, 0}, b2 = {0, 0};
        u64 done = NB == 2 ? 255ull << 8 * (rb ^ (rank & 4)) : pass ? 0 : ~(255ull << 8 * rank);
        u64 nx = ~done, nw = ~done;
        while (done != ~0ull) {
            u64 sv;
            if constexpr (NB == 2) {
                asm volatile("s_mov_b64 %3, exec\n\ts_mov_b64 exec, %6\n\tglobal_load_dwordx4 %0, %4, off sc0 sc1\n\t"
                             "s_mov_b64 exec, %7\n\tglobal_load_dwordx2 %1, %5, off sc0 sc1\n\t"
                             "global_load_dwordx2 %2, %5, off offset:512 sc0 sc1\n\t"
                             "s_mov_b64 exec, %3\n\ts_waitcnt vmcnt(0)"
                             : "+v"(a), "+v"(b), "+v"(b2), "=&s"(sv) : "v"(fx), "v"(fw), "s"(nx), "s"(nw) : "memory");
            } else {
                asm volatile("s_mov_b64 %2, exec\n\ts_mov_b64 exec, %5\n\tglobal_load_dwordx4 %0, %3, off sc0 sc1\n\t"
                             "s_mov_b64 exec, %6\n\tglobal_load_dwordx2 %1, %4, off sc0 sc1\n\t"
                             "s_mov_b64 exec, %2\n\ts_waitcnt vmcnt(0)"
                             : "+v"(a), "+v"(b), "=&s"(sv) : "v"(fx), "v"(fw), "s"(nx), "s"(nw) : "memory");
                b2 = b;
            }
            nx = __builtin_amdgcn_ballot_w64(min(min(a[0], a[1]), min(a[2], a[3])) < call) & ~done;
            nw = __builtin_amdgcn_ballot_w64(min(min(b[0], b[1]), min(b2[0], b2[1])) < call) & ~done;
            u64 ready = ~(nx | nw | done);
            if (!ready && __builtin_amdgcn_s_memrealtime() - t0 > DEADLINE_TICKS) {
                ctl[C_BAD] = 1;
                ready = ~done;
            }
            if (ready) {
                if ((ready >> lane) & 1)
                    ctl[C_ORDER + cnt + __builtin_amdgcn_mbcnt_hi((unsigned)(ready >> 32), __builtin_amdgcn_mbcnt_lo((unsigned)ready, 0))] = lane + 1;
                cnt += __builtin_popcountll(ready);
                done |= ready;
                STAMPN(8 + (nb_dbg < 55 ? nb_dbg : 55), cnt);
                ++nb_dbg;
            }
            if (cnt < NN_STEP - PTAIL) __builtin_amdgcn_s_sleep(PNAP);
            else if (!ready) __builtin_amdgcn_s_sleep(PNAP2);
        }
        }
        STAMP(1);
        return;
    }
    if (wave > 4) {
        const char* const xs = ws + rb * X_RB - X_STEP + lane * 16;
        int n = wave - 5, rn = n, lim = -1;
        unsigned slot = n, need = 0;
        ldsw_t look = ctl + (lane ? C_DONE : C_ORDER) + n;
#pragma unroll 1
        for (;;) {
            const unsigned v = *look;
            if ((int)(__builtin_amdgcn_s_getreg(7 | 23 << 11) & 0xc0000f) <= lim) {
                if (NB == 1 && lim) asm volatile("s_waitcnt vmcnt(24)" ::: "memory");
                else wait_vm0();
                ctl[C_FULL + (rn + NS - 1) % NS] = rn + 1;
                rn += 3;
                lim = lim ? 0 : -1;
                if (rn >= NN_STEP) break;
            }
            const unsigned t = __builtin_amdgcn_readlane(v, 0);
            if (t && (unsigned)__builtin_amdgcn_readlane(v, 1) >= need && (NB == 1 || lim)) {
                const char* wsrc = ws + (NB == 1 && n < 8 ? OFF_W2 + (cg - rank) * 8 * W_STEP : X_BYTES + cg * W_CG) - W_STEP + lane * 16;
                char* l = lds + slot * SLOT;
#pragma unroll
                for (int i = 0; i < 4; ++i) glds4k(xs + t * X_STEP + i * 4096, l + i * 4096);
#pragma unroll
                for (int i = 0; i < 2 * NB; ++i) glds4k(wsrc + t * W_STEP + (i >> 1) * W_CG + (i & 1) * 4096, l + 16384 + i * 4096);
                asm volatile("" ::: "memory");
                lim = lim ? 0 : 0x400008;
                n += 3;
                slot += 3;
                if (slot >= NS) { slot -= NS; need += 4; }
                look = ctl + (lane ? C_DONE + slot : C_ORDER + n);
            } else if (NAP) {
                __builtin_amdgcn_s_sleep(1);
            }
        }
        return;
    }

    const int wm = NB == 2 ? wave >> 1 : wave, wn = NB == 2 ? wave & 1 : 0;
    const int ao = wm * MT * 2048 + (lane & 15) * 128 + ((lane >> 4) ^ ((lane & 15) >> 1)) * 16;
    int slot = 0, n = 0;
    const unsigned l0 = (unsigned)(size_t)(ldsp_t)lds;
    unsigned va[2] = {l0 + ao, l0 + (ao ^ 64)}, vb = l0 + lane * 16 + 16384 + wn * 8192, vc = l0 + LDS_RING + lane * 4, vs = SLOT, one = !lane;
    asm volatile("" : "+v"(vs), "+v"(one));
    char* const sc = lds + LDS_PACK + wave * 640;
    if (NB == 2) *(u32x2*)(sc + lane * 8) = u32x2{0, 0};
    char* qws = uniform_ptr(p.ws[rb ^ (rank & 4)]);
    asm volatile("" : "+s"(qws));
#pragma unroll 1
    for (int pass = NB - 1; pass < 2; ++pass) {
    f32x4 acc[MT][4];
#pragma unroll
    for (int m = 0; m < MT; ++m)
#pragma unroll
        for (int n2 = 0; n2 < 4; ++n2) acc[m][n2] = f32x4{0.f, 0.f, 0.f, 0.f};
    const int nend = pass ? NN_STEP : 8;
    f16x8 a[2][MT], b[2][4];
    auto fetch = [&](int kh, int i) {
        typedef const __attribute__((address_space(3))) f16x8* lp;
        if (i == 0 || i > 4) a[kh][max(i - 4, 0)] = *(lp)(va[kh] + max(i - 4, 0) * 2048);
        else b[kh][i - 1] = *(lp)(vb + kh * 4096 + (i - 1) * 1024);
    };
    unsigned full;
    int d, need, have;
    auto multiply = [&](int kh) {
#pragma unroll
        for (int i = 0; i < 4 * MT; ++i) {
            acc[i >> 2][i & 3] = __builtin_amdgcn_mfma_f32_16x16x32_f16(a[kh][i >> 2], b[kh][i & 3], acc[i >> 2][i & 3], 0, 0, 0);
            __builtin_amdgcn_sched_barrier(0);
            if (i < MT + 4) fetch(kh ^ 1, i);
            if (kh == 0) {
                if (i == 0) full = *(ldsw_t)(vc + C_FULL * 4);
                if (i == 1) asm volatile("s_cmp_eq_u32 %1, %2\n\ts_cselect_b32 %0, %3, 1" : "=s"(d) : "s"(slot), "n"(NS - 1), "n"(1 - NS) : "scc");
                if (i == 2) asm volatile("s_add_i32 %0, %0, %2\n\ts_add_i32 %1, %1, 1" : "+s"(slot), "+s"(n) : "s"(d) : "scc");
                if (i == 3) asm volatile("s_cmp_lt_i32 %1, %2\n\ts_cselect_b32 %0, %1, -1" : "=s"(need) : "s"(n), "s"(nend) : "scc");
                if (i == 4) asm volatile("v_mad_i32_i24 %0, %1, %2, %0" : "+v"(va[0]) : "s"(d), "v"(vs));
                if (i == 5) asm volatile("v_mad_i32_i24 %0, %1, %2, %0" : "+v"(vb) : "s"(d), "v"(vs));
                if (i == MT + 4) {
                    asm volatile("" ::: "memory");
                    __hip_atomic_fetch_add((__attribute__((address_space(3))) unsigned*)(vc + C_DONE * 4), one, __ATOMIC_RELAXED, __HIP_MEMORY_SCOPE_WORKGROUP);
                    have = __builtin_amdgcn_readfirstlane(full);
                }
            } else {
                if (i == MT + 4) asm volatile("v_mad_i32_i24 %0, %2, %3, %0\n\tv_lshl_add_u32 %1, %2, 2, %1" : "+v"(va[1]), "+v"(vc) : "s"(d), "v"(vs));
            }
            __builtin_amdgcn_sched_barrier(0);
        }
    };
    lds_wait(ctl + C_FULL + (slot ? slot : NS) - 1, n + 1);
#pragma unroll
    for (int i = 0; i < MT + 4; ++i) { fetch(0, i); __builtin_amdgcn_sched_barrier(0); }
#pragma unroll 1
    do {
        multiply(0);
        if (__builtin_expect(have <= need, 0)) lds_wait((ldsw_t)(vc + C_FULL * 4), n + 1);
        asm volatile("" ::: "memory");
        multiply(1);
        if ((n & 3) == 0) STAMP(7 + (n >> 2));
    } while (need >= 0);
    STAMP(1);

    const bool bad = lds_get(ctl + C_BAD);
    if constexpr (NB == 1) {
        char* o = (char*)(p.out + (size_t)(rb * 128 + wave * 32 + (lane >> 4) * 4) * NN + ((rank >> 2) ^ pass ^ 1) * 2048 + cg * 64 + (lane & 15) * 4);
        if (pass) {
            const float one = bad ? __builtin_nanf("") : 1.f;
#pragma unroll
            for (int r = 0; r < 8; r += 2) {
                f32x4 u[2];
#pragma unroll
                for (int c = 0; c < 4; ++c) {
                    const float y0 = acc[r >> 2][c][r & 3], y1 = acc[r >> 2][c][(r & 3) + 1];
                    const f32x2 g = gate(f32x2{y0, y1}, one);
                    float z0 = y0 * g[0], z1 = y1 * g[1];
                    asm("" : "+v"(z0));
                    asm("" : "+v"(z1));
                    u[0][c] = z0;
                    u[1][c] = z1;
                }
                store16_sys(o + ((r >> 2) * 16 + (r & 3)) * NN * 4, __builtin_bit_cast(u32x4, u[0]));
                store16_sys(o + ((r >> 2) * 16 + (r & 3) + 1) * NN * 4, __builtin_bit_cast(u32x4, u[1]));
                __builtin_amdgcn_sched_barrier(0);
            }
        } else {
#pragma unroll
            for (int r = 0; r < 8; ++r)
                store16_sys(o + ((r >> 2) * 16 + (r & 3)) * NN * 4,
                            __builtin_bit_cast(u32x4, f32x4{acc[r >> 2][0][r & 3], acc[r >> 2][1][r & 3], acc[r >> 2][2][r & 3], acc[r >> 2][3][r & 3]}));
            wait_vm0();
            if (lds_add(ctl + C_PT, bad ? 16 : 1) == 3 && !bad) flag_sys(ws, OFF_FE + (((rank & 4) ^ 4) + rb) * 1024 + (160 + cg) * 4, call);
        }
    } else {
        const unsigned o = OFF_OX + (((rb - 4) * 16 + (cg >> 1)) * 4 + wave) * 4 * PO + lane * 16;
        const unsigned f = po + OFF_FD + ((rank & 3) * 16 + (cg >> 1)) * 64 + wave * 4;
        if (bad) return;
        unsigned es = 0;
#pragma unroll
        for (int m = 0; m < 4; ++m) es += (__float_as_uint(acc[m][m][m]) >> 23) & 255;
        const unsigned eb = min(max((wave_sum(es) - 1068) >> 8, 0), 248);
        unsigned ht[4][16], pt[4][4], xt[4];
        auto prepare = [&](int m) {
#pragma unroll
            for (int c = 0; c < 16; ++c) ht[m][c] = __float_as_uint(acc[m][c >> 2][c & 3]) - (eb << 23);
            unsigned nz[2] = {0, 0};
#pragma unroll
            for (int k = 0; k < 4; ++k) {
                pt[m][k] = __builtin_amdgcn_perm(__builtin_amdgcn_perm(ht[m][4 * k + 3], ht[m][4 * k + 2], 0x07030703u),
                                                 __builtin_amdgcn_perm(ht[m][4 * k + 1], ht[m][4 * k], 0x07030703u), 0x05040100u) & 0x7c7c7c7cu;
                nz[k >> 1] = __builtin_amdgcn_udot4((pt[m][k] + 0x7c7c7c7cu) & 0x80808080u, k & 1 ? 0x80402010u : 0x08040201u, nz[k >> 1], false);
            }
            xt[m] = nz[0] | nz[1] << 8;
        };
        prepare(0);
        TICKS(tq[12]);
#pragma unroll
        for (int m = 0; m < 4; ++m) {
            unsigned* const h = ht[m]; const unsigned* const pl = pt[m];
            unsigned cnt = 0, cv = 0, t0, t1, t2, n; u64 sv;
            asm volatile("s_mov_b64 %7, exec\n\tv_cmpx_ne_u32_e32 vcc, 0, %0\n\ts_cbranch_execz 2f\n1:\n\t"
                         "v_ffbl_b32_e32 %3, %0\n\tv_add_u32_e32 %4, -1, %0\n\tv_and_b32_e32 %0, %0, %4\n\t"
                         "v_cmp_lt_u32_e32 vcc, 14, %3\n\tv_add_u32_e32 %3, %3, %10\n\tv_and_b32_e32 %5, 7, %3\n\t"
                         "v_perm_b32 %4, %9, %8, %5\n\tv_perm_b32 %5, %13, %12, %5\n\tv_cndmask_b32_e32 %4, %4, %5, vcc\n\t"
                         "v_lshl_or_b32 %3, %4, 8, %3\n\t" LIST_TAIL
                         : "+v"(xt[m]), "+v"(cv), "+s"(cnt), "=&v"(t0), "=&v"(t1), "=&v"(t2), "=&s"(n), "=&s"(sv)
                         : "v"(pl[0]), "v"(pl[1]), "v"(lane * 16 - 7), "s"((unsigned)(size_t)(ldsp_t)(sc + m * 128)), "v"(pl[2]), "v"(pl[3]) : "memory", "vcc", "scc");
            TICKS(tq[3 * m]);
            unsigned ent = *(const unsigned short*)(sc + m * 128 + lane * 2);
            __builtin_amdgcn_sched_barrier(0);
            if (m < 3) prepare(m + 1);
            __builtin_amdgcn_sched_barrier(0);
            if (lane == 63) ent = eb | (cnt > 63 ? 0x8000 : 0);
#pragma unroll
            for (int k = 0; k < 2; ++k) {
                const unsigned d = bfi(0x7c000000u, ent << (26 - 8 * k), h[14 + k]);
#pragma unroll
                for (int j = 0; j < 6; ++j) h[7 * k + j] = bfi(0x7c000000u, d << (26 - 5 * j), h[7 * k + j]);
                h[7 * k + 6] = bfi(0x7c000000u, __builtin_amdgcn_alignbit(ent >> (8 * k + 5), d, 4), h[7 * k + 6]);
            }
            if (cnt > 63)
                asm volatile("global_store_dwordx4 %0, %1, %2 offset:3584 sc0 sc1\n\ts_nop 1" :: "v"(o + m * PO), "v"(u32x4{pl[0], pl[1], pl[2], pl[3]}), "s"(ws) : "memory");
            asm volatile("global_store_dwordx4 %1, %3, %7 sc0 sc1\n\tglobal_store_dwordx4 %1, %4, %7 offset:1024 sc0 sc1\n\t"
                         "global_store_dwordx4 %1, %5, %7 offset:2048 sc0 sc1\n\ts_mov_b64 %0, exec\n\ts_mov_b64 exec, %8\n\t"
                         "global_store_dwordx4 %2, %6, %7 offset:3072 sc0 sc1\n\ts_mov_b64 exec, %0\n\ts_nop 1"
                         : "=&s"(sv) : "v"(o + m * PO), "v"(o + m * PO - lane * 8), "v"(u32x4{h[0], h[1], h[2], h[3]}), "v"(u32x4{h[4], h[5], h[6], h[7]}),
                           "v"(u32x4{h[8], h[9], h[10], h[11]}),
                           "v"(u32x4{h[12], h[13], (unsigned)__builtin_amdgcn_mov_dpp(h[12], 0xf5, 0xf, 0xf, true), (unsigned)__builtin_amdgcn_mov_dpp(h[13], 0xf5, 0xf, 0xf, true)}),
                           "s"(ws), "s"(0x5555555555555555ull) : "memory");
            TICKS(tq[3 * m + 1]);
        }
#pragma unroll
        for (int m = 0; m < 4; ++m) {
            asm volatile("s_waitcnt vmcnt(%0)" :: "n"(12 - 3 * m) : "memory");
            flag_sys(qws, f + m * 16, call);
            if (m < 3) TICKS(tq[3 * m + 2]);
        }
        TICKS(tq[11]);
    }
    }
    STAMP(2);
}

__global__ void __launch_bounds__(BLOCK, 1) fused_kernel(Params p) {
    extern __shared__ __attribute__((aligned(16))) char lds[];
    if (WARM && (blockIdx.x >> 3) == 24 && threadIdx.x < WARM) {
        unsigned t;
        asm volatile("global_load_dword %0, %1, %2\n\ts_waitcnt vmcnt(0)" : "=v"(t) : "v"(threadIdx.x * 128), "s"(__builtin_amdgcn_s_getpc()) : "memory");
    }
    const unsigned call = pack_role(p, lds);
    const int xcd = blockIdx.x & 7, lb = blockIdx.x >> 3;
    if (lb < 8) {
        tile_role<2>(p, lds, 4 + (lb & 3), xcd * 4 + (lb >> 2) * 2, call);
    } else if (lb < 24) {
        const int k = lb - 8;
        tile_role<1>(p, lds, k & 3, xcd * 4 + (k >> 2), call);
    } else if (lb > 24) {
        pull_role(p, lds, lb - 24, xcd, call);
    } else if (xcd == 0 && threadIdx.x < 64) {
        plan_role(p, lds, call);
    }
}

}

extern "C" void kernel_launch(int rank, int nranks, ncclComm_t comm,
                              hipStream_t stream,
                              void* const* d_in, const int* in_sizes, int n_in,
                              void* d_out, int out_size,
                              const int* in_dtypes, int out_dtype,
                              void* const* d_out_all,
                              void* const* d_ws_all, size_t ws_bytes) {
    if (nranks != NR || (unsigned)rank >= (unsigned)NR || ws_bytes < WS_NEED || n_in < 2 ||
        in_sizes[0] != MM * KS || in_sizes[1] != KS * NN || out_size != (MM / NR) * NN) {
        fprintf(stderr, "kernel_launch: unexpected shapes\n");
        return;
    }
    HIP_CHECK(hipFuncSetAttribute((const void*)fused_kernel, hipFuncAttributeMaxDynamicSharedMemorySize, LDS_BYTES));
    Params p;
    p.x = (const float*)d_in[0];
    p.w = (const float*)d_in[1];
    p.out = (float*)d_out;
    for (int i = 0; i < NR; ++i) p.ws[i] = (char*)d_ws_all[i];
    p.rank = rank; p.pad = 0;
    p.lws = p.ws[rank];
    for (int i = 0; i < 2; ++i) { p.ntab[i] = NTAB[i]; for (int j = 0; j < 256; ++j) p.tab[i * 256 + j] = TAB[i][j]; }
    fused_kernel<<<GRID, BLOCK, LDS_BYTES, stream>>>(p);
    HIP_CHECK(hipGetLastError());
}
```
